```python
import numpy as np

kernel_name = "gpu_gemm_a2a_m4096_k4096_n16384_e5m2e5m2_dequant"


ABSMAX_RTOL = 0.05

COMPUTE_DTYPE = "float32"
EPILOGUE = "dequant"
IN_QUANT = None
OUT_QUANT = None
IN_FP8 = "e5m2e5m2"
IN_DTYPES = {"x": "e5m2", "w_mat": "e5m2",
             "scale_x": "float32", "scale_w": "float32"}
WORLD_SIZE = 8

def _snap_fp8(x, mant_bits, max_abs, min_normal):
    a = np.abs(x.astype(np.float64))
    _m, _e = np.frexp(a)
    q = np.where(a >= min_normal, np.ldexp(1.0, _e - 1 - mant_bits),
                 min_normal * 2.0 ** -mant_bits)
    snapped = np.minimum(np.round(a / q) * q, max_abs)
    return (np.sign(x) * snapped).astype(np.float32)

def setup_inputs():

    x = _snap_fp8(np.random.randn(4096, 4096).astype(np.float32), 2, 57344.0, 2.0 ** -14)
    w_mat = _snap_fp8(np.random.randn(4096, 16384).astype(np.float32), 2, 57344.0, 2.0 ** -14)
    scale_x = (0.02 * (1.0 + np.random.rand(1))).astype(np.float32)
    scale_w = (0.02 * (1.0 + np.random.rand(1))).astype(np.float32)
    return {"x": x, "w_mat": w_mat, "scale_x": scale_x, "scale_w": scale_w}

def reference(x, w_mat, scale_x, scale_w):
    acc = (x @ w_mat).astype(np.float32)
    y = (acc * (scale_x * scale_w)).astype(np.float32)
    return y
```

```cpp
#include <hip/hip_runtime.h>
#include <rccl/rccl.h>
#include <cstdio>
#include <cstdint>
#include <type_traits>

#define HIP_CHECK(expr) do { hipError_t _e = (expr); if (_e != hipSuccess) { \
    fprintf(stderr, "HIP error %s at %s:%d\n", hipGetErrorString(_e), __FILE__, __LINE__); \
    return; } } while (0)

typedef unsigned v4u __attribute__((ext_vector_type(4)));
typedef int v8i __attribute__((ext_vector_type(8)));
typedef int v2i __attribute__((ext_vector_type(2)));
typedef float v4f __attribute__((ext_vector_type(4)));
#define LAS __attribute__((address_space(3)))
#define SB __builtin_amdgcn_sched_barrier(0)

namespace {

constexpr int NR = 8;
constexpr int M = 4096, K = 4096, NFULL = 16384, NS = 2048, MS = 512;
constexpr unsigned SHARD = MS * K;
constexpr unsigned PIECE = 16 * K;
constexpr int NPIECE = MS / 16;
constexpr int NRG = 7;
constexpr int NWG = 256;
constexpr int LDS_BYTES = 131072;

constexpr unsigned OFF_SRC = 0;
constexpr unsigned OFF_DST = 2 * SHARD;
constexpr unsigned OFF_FLS = 9 * SHARD;
constexpr unsigned OFF_FLP = OFF_FLS + NR * NWG * 128;
constexpr unsigned OFF_CNT = OFF_FLP + NRG * 16 * 128;
constexpr unsigned OFF_TMO = OFF_CNT + NWG * 128;
constexpr unsigned OFF_DBG = OFF_TMO + 128;
constexpr unsigned WS_USED = OFF_DBG + NWG * 64 * 8;
constexpr unsigned long long DEADLINE = 30000000ull;

struct Params {
  char* ws[NR];
  char* wsl;
  const void* tab;
  const char* x;
  const char* w;
  const float* sx;
  const float* sw;
  float* out;
  int rank;
  int pad;
};

#define STAMP(i) do {} while (0)

__device__ __forceinline__ void mfma0(v4f& acc, const v8i& w, const v8i& x) {
  asm volatile("v_mfma_f32_16x16x128_f8f6f4 %0, %1, %2, 0 cbsz:1 blgp:1" : "=&v"(acc) : "a"(w), "v"(x));
}
__device__ __forceinline__ void mfma(v4f& acc, const v8i& w, const v8i& x) {
  asm volatile("v_mfma_f32_16x16x128_f8f6f4 %0, %1, %2, %0 cbsz:1 blgp:1" : "+v"(acc) : "a"(w), "v"(x));
}
__device__ __forceinline__ unsigned ld_sys(const void* p) {
  unsigned v;
  asm volatile("global_load_dword %0, %1, off sc0 sc1\n\ts_waitcnt vmcnt(0)" : "=&v"(v) : "v"(p) : "memory");
  return v;
}
__device__ __forceinline__ void st_sys(void* p, unsigned v) {
  asm volatile("global_store_dword %0, %1, off sc0 sc1" :: "v"(p), "v"(v) : "memory");
}
__device__ __forceinline__ bool spin_ge(const void* p, unsigned target, char* wsl, unsigned call) {
  unsigned long long t0 = __builtin_amdgcn_s_memrealtime();
  for (unsigned n = 0;; ++n) {
    unsigned f = ld_sys(p);
    if (__builtin_amdgcn_ballot_w64(f < target) == 0) return true;
    if ((n & 63) == 63) {
      unsigned t = ld_sys(wsl + OFF_TMO);
      if (__builtin_amdgcn_readfirstlane(t) == call || __builtin_amdgcn_s_memrealtime() - t0 > DEADLINE) {
        st_sys(wsl + OFF_TMO, call);
        return false;
      }
    }
    __builtin_amdgcn_s_sleep(1);
  }
}

__global__ void __launch_bounds__(256, 1) fused_ag_gemm(Params P) {
  extern __shared__ __attribute__((aligned(16))) char smem[];
  const int b = blockIdx.x, xcd = b & 7, slot = b >> 3;
  const int wave = __builtin_amdgcn_readfirstlane(threadIdx.x >> 6), lane = threadIdx.x & 63;
  const int g = lane >> 4, m = lane & 15;
  const unsigned rank = (unsigned)P.rank & 7u;
  char* const wsl = P.wsl;
  STAMP(0);

  unsigned call;
  {
    const unsigned fr = (unsigned)b * 4 + wave;
    const char* xs = P.x + (size_t)((fr >> 5) * 16 + m) * K + (fr & 31) * 128 + g * 16;
    v4u lo, hi; unsigned c; char* peer;
    asm volatile("global_load_dwordx4 %0, %4, off\n\t"
                 "global_load_dwordx4 %1, %4, off offset:64\n\t"
                 "global_load_dword %2, %5, off sc0 sc1\n\t"
                 "global_load_dwordx2 %3, %6, off\n\t"
                 "s_waitcnt vmcnt(0)"
                 : "=&v"(lo), "=&v"(hi), "=&v"(c), "=&v"(peer)
                 : "v"(xs), "v"(wsl + OFF_CNT + b * 128), "v"((const char*)P.tab + (lane & 7) * 8) : "memory");
    call = __builtin_amdgcn_readfirstlane(c) + 1;
    char* d = wsl + OFF_SRC + (call & 1) * SHARD + fr * 2048 + lane * 16;
    asm volatile("global_store_dwordx4 %0, %1, off sc0 sc1\n\t"
                 "global_store_dwordx4 %0, %2, off offset:1024 sc0 sc1\n\t"
                 "s_waitcnt vmcnt(0)"
                 :: "v"(d), "v"(lo), "v"(hi) : "memory");
    __builtin_amdgcn_s_barrier();
    if (wave == 0) {
      char* dst = lane < NR ? peer + OFF_FLS + (rank * NWG + b) * 128 : wsl + OFF_CNT + b * 128;
      if (lane <= NR) st_sys(dst, call);
    }
  }
  STAMP(1);

  if (slot < 4) {
    const int u = slot * 8 + xcd;
    if (u >= 4 * NRG) return;
    __builtin_amdgcn_s_waitcnt(0x0F70);
    const int pi = u % NRG, v = (u / NRG) * 4 + wave;
    const unsigned p = (rank + 1 + pi) & 7u;
    if (!spin_ge(wsl + OFF_FLS + (p * NWG + (lane & 31) * 8 + (v >> 1)) * 128, call, wsl, call)) return;
    STAMP(2);
    const char* rsrc = P.ws[p] + OFF_SRC + (call & 1) * SHARD;
    char* rdst = wsl + OFF_DST + pi * SHARD;
    char* flag = wsl + OFF_FLP + (pi * 16 + v) * 128;
    const unsigned voff = v * 4096 + lane * 16;
    v4u b0[4], b1[4];
    #define LD4(B, so) asm volatile("global_load_dwordx4 %0, %4, %5 sc0 sc1\n\t" \
                                    "global_load_dwordx4 %1, %4, %5 offset:1024 sc0 sc1\n\t" \
                                    "global_load_dwordx4 %2, %4, %5 offset:2048 sc0 sc1\n\t" \
                                    "global_load_dwordx4 %3, %4, %5 offset:3072 sc0 sc1" \
                                    : "=&v"(B[0]), "=&v"(B[1]), "=&v"(B[2]), "=&v"(B[3]) : "v"(voff), "s"(rsrc + (so)) : "memory")
    #define ST4(B, so, N) asm volatile("s_waitcnt vmcnt(" #N ")\n\t" \
                                    "global_store_dwordx4 %4, %0, %5 sc0 sc1\n\t" \
                                    "global_store_dwordx4 %4, %1, %5 offset:1024 sc0 sc1\n\t" \
                                    "global_store_dwordx4 %4, %2, %5 offset:2048 sc0 sc1\n\t" \
                                    "global_store_dwordx4 %4, %3, %5 offset:3072 sc0 sc1\n\ts_nop 1" \
                                    :: "v"(B[0]), "v"(B[1]), "v"(B[2]), "v"(B[3]), "v"(voff), "s"(rdst + (so)) : "memory")
    const unsigned base = call * 64;
    LD4(b0, 0u);
    LD4(b1, PIECE);
    ST4(b0, 0u, 4);
    st_sys(flag, base);
    for (int j = 1; j < NPIECE - 1; j += 2) {
      LD4(b0, (unsigned)(j + 1) * PIECE);
      ST4(b1, (unsigned)j * PIECE, 9);
      asm volatile("s_waitcnt vmcnt(9)" ::: "memory");
      st_sys(flag, base + j);
      LD4(b1, (unsigned)(j + 2) * PIECE);
      ST4(b0, (unsigned)(j + 1) * PIECE, 9);
      asm volatile("s_waitcnt vmcnt(9)" ::: "memory");
      st_sys(flag, base + j + 1);
    }
    ST4(b1, (unsigned)(NPIECE - 1) * PIECE, 5);
    asm volatile("s_waitcnt vmcnt(5)" ::: "memory");
    st_sys(flag, base + NPIECE - 1);
    asm volatile("s_waitcnt vmcnt(1)" ::: "memory");
    st_sys(flag, base + NPIECE);
    STAMP(3);
    return;
  }

  const int i = slot - 4;
#define GMAP 3
  int rg, cb;
  if (GMAP == 0) { rg = i % NRG; cb = xcd * 4 + i / NRG; }
  else { const int q = (xcd & 1) * 28 + i; rg = q >> 3; cb = (xcd >> 1) * 8 + (q & 7); }
  const int rot = (rg + cb) & 7;

  v8i Wf[8][4];
  {
    __amdgpu_buffer_rsrc_t rw = __builtin_amdgcn_make_buffer_rsrc((void*)P.w, 0, K * NFULL, 0x00020000);
    char* wl = smem + wave * 32768;
    const unsigned wl_addr = (unsigned)(size_t)(LAS char*)wl;
    const int r16 = lane >> 2;
    const unsigned vb = (unsigned)(wave * 1024 + r16) * NFULL + rank * NS + cb * 64;
    const unsigned v0 = vb + (((lane & 3) ^ ((r16 >> 2) & 1)) << 4);
    const unsigned v1 = vb + (((lane & 3) ^ (((r16 >> 2) & 1) | 2)) << 4);
    auto issue = [&](int t) {
      #pragma unroll
      for (int q = 0; q < 8; ++q)
        asm volatile("s_mov_b32 m0, %0\n\ts_nop 0\n\tbuffer_load_dwordx4 %1, %2, %3 offen lds"
                     :: "s"(wl_addr + (t & 3) * 8192 + q * 1024), "v"((q & 1) ? v1 : v0), "s"(rw),
                        "s"((unsigned)((((t + rot) & 7) * 128 + q * 16) * NFULL)) : "memory");
    };
    issue(0); issue(1); issue(2); issue(3);
    const int q8 = (lane & 15) >> 1, p = lane & 1;
    #pragma unroll
    for (int t = 0; t < 8; ++t) {
      if (t <= 4) asm volatile("s_waitcnt vmcnt(24)" ::: "memory");
      else if (t == 5) asm volatile("s_waitcnt vmcnt(16)" ::: "memory");
      else if (t == 6) asm volatile("s_waitcnt vmcnt(8)" ::: "memory");
      else asm volatile("s_waitcnt vmcnt(0)" ::: "memory");
      #pragma unroll
      for (int c = 0; c < 4; ++c) {
        v2i r[4];
        #pragma unroll
        for (int he = 0; he < 4; ++he) {
          const int row = 64 * (he >> 1) + 16 * g + 8 * (he & 1) + q8;
          const int f = ((row >> 2) & 1) | (((row >> 4) & 1) << 1);
          r[he] = __builtin_amdgcn_ds_read_tr8_b64_v2i32((LAS v2i*)(wl + (t & 3) * 8192 + row * 64 + ((c ^ f) << 4) + p * 8));
        }
        v8i w = {r[0].x, r[0].y, r[1].x, r[1].y, r[2].x, r[2].y, r[3].x, r[3].y};
        Wf[t][c] = w;
      }
      asm volatile("s_waitcnt lgkmcnt(0)" : "+a"(Wf[t][0]), "+a"(Wf[t][1]), "+a"(Wf[t][2]), "+a"(Wf[t][3]) :: "memory");
      if (t < 4) issue(t + 4);
    }
  }
  STAMP(2);

  float scale = P.sx[0] * P.sw[0];
  const int nown = (NPIECE - rg + NRG - 1) / NRG;
  const int np = nown + NPIECE;
  const unsigned src = (rank + 1 + rg) & 7u;
  {
    int jj = rg + NRG * (lane >> 1); if (jj >= NPIECE) jj = rg;
    if (!spin_ge(wsl + OFF_FLS + (rank * NWG + jj * 8 + 2 * wave + (lane & 1)) * 128, call, wsl, call)) scale = __builtin_nanf("");
  }
  auto xoff = [&](int n) -> unsigned {
    return n < nown ? OFF_SRC + (call & 1) * SHARD + (unsigned)(rg + NRG * n) * PIECE
                    : (n < np ? OFF_DST + rg * SHARD + (unsigned)(n - nown) * PIECE : OFF_FLS);
  };
  auto orow = [&](int n) -> unsigned {
    return n < nown ? rank * MS + (unsigned)(rg + NRG * n) * 16 : src * MS + (unsigned)(n - nown) * 16;
  };
  __amdgpu_buffer_rsrc_t rs = __builtin_amdgcn_make_buffer_rsrc((void*)wsl, 0, (int)OFF_FLS, 0x00020000);
  __amdgpu_buffer_rsrc_t rf = __builtin_amdgcn_make_buffer_rsrc((void*)(wsl + OFF_FLP), 0, NRG * 16 * 128, 0x00020000);
  __amdgpu_buffer_rsrc_t ro = __builtin_amdgcn_make_buffer_rsrc((void*)P.out, 0, M * NS * 4, 0x00020000);
  constexpr int AUX = 17;
  const unsigned voff = wave * 16384 + lane * 16;
  const unsigned foff = (rg * 16 + wave * 4 + (lane & 3)) * 128;
  v4u xb[16];
  {
    const unsigned sb = xoff(0);
    #pragma unroll
    for (int t = 0; t < 16; ++t) xb[t] = __builtin_amdgcn_raw_buffer_load_b128(rs, voff + (t & 1) * 1024, sb + ((((t >> 1) + rot) & 7) << 11), AUX);
  }
  __builtin_amdgcn_s_barrier();

  const unsigned wr_off = wave * 4096 + m * 256;
  const int rrow = 4 * wave + g;
  const unsigned rd_off = rrow * 256 + ((m ^ rrow) << 4);
  const unsigned ooff = (rrow * NS + cb * 64 + m * 4) * 4;
  v4f acc[2][4];
  unsigned fl = 0;
  bool alive = true;
  auto step = [&](auto Bc, int n, bool prev, bool cur) {
    constexpr int B = decltype(Bc)::value;
    const unsigned sb = xoff(n + 1), sbp = xoff(n);
    char* rb = smem + (B ^ 1) * 16384;
    v4f s0, s1, s2, s3, s;
    #pragma unroll
    for (int t = 0; t < 8; ++t) {
      const v4u lo = xb[2 * t], hi = xb[2 * t + 1];
      const v8i x = {(int)lo.x, (int)lo.y, (int)lo.z, (int)lo.w, (int)hi.x, (int)hi.y, (int)hi.z, (int)hi.w};
      #pragma unroll
      for (int c = 0; c < 4; ++c) {
        const int gap = t * 4 + c;
        if (cur) { if (t == 0) mfma0(acc[B][c], Wf[t][c], x); else mfma(acc[B][c], Wf[t][c], x); }
        SB;
        if (cur && gap == 3) {
          if (n + 1 >= nown && n + 1 < np && alive) {
            const unsigned target = call * 64 + (unsigned)(n + 2 - nown);
            if (__builtin_amdgcn_ballot_w64(fl < target) != 0) {
              if (!spin_ge(wsl + OFF_FLP + foff, target, wsl, call)) { alive = false; scale = __builtin_nanf(""); }
            }
          }
        }
        if (cur && (c == 0 || c == 2)) {
          if (t > 0) xb[2 * (t - 1) + (c >> 1)] = __builtin_amdgcn_raw_buffer_load_b128(rs, voff + (c >> 1) * 1024, sb + (((t - 1 + rot) & 7) << 11), AUX);
          else if (prev) xb[14 + (c >> 1)] = __builtin_amdgcn_raw_buffer_load_b128(rs, voff + (c >> 1) * 1024, sbp + (((7 + rot) & 7) << 11), AUX);
        }
        if (cur && gap == 13) fl = __builtin_amdgcn_raw_buffer_load_b32(rf, foff, 0, AUX);
        if (prev) {
          if (gap == 0 && !cur) asm volatile("s_nop 7\n\ts_nop 7" ::: "memory");
          if (gap == 5 || gap == 7 || gap == 9 || gap == 11) { const int cc = (gap - 5) >> 1; *(LAS v4f*)(rb + wr_off + (((4 * cc + g) ^ m) << 4)) = acc[B ^ 1][cc]; }
          if (gap == 15) { asm volatile("s_waitcnt lgkmcnt(0)" ::: "memory"); __builtin_amdgcn_s_barrier(); asm volatile("" ::: "memory"); }
          if (gap == 17) { s0 = *(LAS v4f*)(rb + rd_off); s1 = *(LAS v4f*)(rb + 4096 + rd_off); }
          if (gap == 19) { s2 = *(LAS v4f*)(rb + 8192 + rd_off); s3 = *(LAS v4f*)(rb + 12288 + rd_off); }
          if (gap == 23) { for (int e = 0; e < 4; ++e) asm("v_add_f32 %0, %1, %2" : "=v"(s[e]) : "v"(s0[e]), "v"(s1[e])); }
          if (gap == 25) { for (int e = 0; e < 4; ++e) asm("v_add_f32 %0, %1, %2" : "=v"(s2[e]) : "v"(s2[e]), "v"(s3[e])); }
          if (gap == 27) { for (int e = 0; e < 4; ++e) asm("v_add_f32 %0, %1, %2" : "=v"(s[e]) : "v"(s[e]), "v"(s2[e])); }
          if (gap == 29) { for (int e = 0; e < 4; ++e) asm("v_mul_f32 %0, %1, %2" : "=v"(s[e]) : "v"(scale), "v"(s[e])); }
          if (gap == 31) __builtin_amdgcn_raw_buffer_store_b128(__builtin_bit_cast(v4u, s), ro, ooff, orow(n - 1) * (NS * 4), 0);
        }
        SB;
      }
    }
  };
  std::integral_constant<int, 0> I0; std::integral_constant<int, 1> I1;
  step(I0, 0, false, true);
  int n = 1;
  for (; n + 1 < np; n += 2) {
    step(I1, n, true, true); step(I0, n + 1, true, true);
  }
  if (n < np) { step(I1, n, true, true); step(I0, n + 1, true, false); }
  else step(I1, n, true, false);
  STAMP(3);
}

}

extern "C" void kernel_launch(int rank, int nranks, ncclComm_t comm, hipStream_t stream,
                              void* const* d_in, const int* in_sizes, int n_in,
                              void* d_out, int out_size,
                              const int* in_dtypes, int out_dtype,
                              void* const* d_out_all,
                              void* const* d_ws_all, size_t ws_bytes) {
  (void)comm; (void)in_dtypes; (void)out_dtype; (void)d_out_all;
  if (nranks != NR || n_in != 4 || (unsigned)rank >= (unsigned)NR || ws_bytes < WS_USED ||
      in_sizes[0] != MS * K || in_sizes[1] != K * NFULL || out_size != M * NS) {
    fprintf(stderr, "kernel_launch: unexpected problem shape\n");
    return;
  }
  HIP_CHECK(hipFuncSetAttribute((const void*)fused_ag_gemm, hipFuncAttributeMaxDynamicSharedMemorySize, LDS_BYTES));
  Params P;
  for (int r = 0; r < NR; ++r) P.ws[r] = (char*)d_ws_all[r];
  P.wsl = P.ws[rank];
  P.tab = (const void*)d_ws_all;
  P.x = (const char*)d_in[0];
  P.w = (const char*)d_in[1];
  P.sx = (const float*)d_in[2];
  P.sw = (const float*)d_in[3];
  P.out = (float*)d_out;
  P.rank = rank;
  P.pad = 0;
  fused_ag_gemm<<<NWG, 256, LDS_BYTES, stream>>>(P);
  HIP_CHECK(hipGetLastError());
}
```

```cpp
#include <hip/hip_runtime.h>
#include <rccl/rccl.h>
#include <cstdio>
#include <cstdint>
#include <type_traits>

#define HIP_CHECK(expr) do { hipError_t _e = (expr); if (_e != hipSuccess) { \
    fprintf(stderr, "HIP error %s at %s:%d\n", hipGetErrorString(_e), __FILE__, __LINE__); \
    return; } } while (0)

typedef unsigned v4u __attribute__((ext_vector_type(4)));
typedef int v8i __attribute__((ext_vector_type(8)));
typedef int v2i __attribute__((ext_vector_type(2)));
typedef float v4f __attribute__((ext_vector_type(4)));
#define LAS __attribute__((address_space(3)))
#define SB __builtin_amdgcn_sched_barrier(0)

namespace {

constexpr int NR = 8;
constexpr int M = 4096, K = 4096, NFULL = 16384, NS = 2048, MS = 512;
constexpr unsigned SHARD = MS * K;
constexpr unsigned PIECE = 16 * K;
constexpr int NPIECE = MS / 16;
constexpr int NRG = 7;
constexpr int NWG = 256;
constexpr int LDS_BYTES = 131072 + 1024;

#define NPUSH 2
#define KDRAIN 6
constexpr unsigned PKU = 3072;
constexpr int LCAP = 62;

constexpr unsigned OFF_RAW = 0;
constexpr unsigned OFF_DST = SHARD;
constexpr unsigned OFF_PSH = 8 * SHARD;
constexpr unsigned PSH_PAR = SHARD;
constexpr unsigned OFF_END = 10 * SHARD;
constexpr unsigned OFF_PK  = OFF_END;
constexpr unsigned PK_PAR = SHARD, PK_LS = 512 * PKU, PK_PAD = PK_LS + 512 * 128;
constexpr unsigned OFF_PL  = 12 * SHARD;
constexpr unsigned PL_PAR = 512 * 1024;
constexpr unsigned OFF_FLX = 13 * SHARD;
constexpr unsigned OFF_FGO = OFF_FLX + 128 * 128;
constexpr unsigned OFF_FLP = OFF_FGO + NRG * 4 * 128;
constexpr unsigned OFF_CNT = OFF_FLP + NRG * 4 * 32 * 128;
constexpr unsigned OFF_TMO = OFF_CNT + NWG * 128;
constexpr unsigned OFF_PAD = OFF_TMO + 128;
constexpr unsigned OFF_DBG = OFF_PAD + 64 * 128;
constexpr unsigned WS_USED = OFF_DBG + NWG * 64 * 4;
constexpr unsigned long long DEADLINE = 30000000ull;
static_assert(NPUSH >= 0 && NPUSH <= 3 && 7 * 4 * PIECE <= PSH_PAR && PK_PAD + 64 * 128 <= PK_PAR, "layout");

struct Params {
  char* ws[NR];
  char* wsl;
  const char* x;
  const char* w;
  const float* sx;
  const float* sw;
  float* out;
  int rank;
  int pad;
};

#define STAMP(i) do {} while (0)
#define STAMP_OUT(w) do {} while (0)

__device__ __forceinline__ void mfma0(v4f& acc, const v8i& w, const v8i& x) {
  asm volatile("v_mfma_f32_16x16x128_f8f6f4 %0, %1, %2, 0 cbsz:1 blgp:1" : "=&v"(acc) : "a"(w), "v"(x));
}
__device__ __forceinline__ void mfma(v4f& acc, const v8i& w, const v8i& x) {
  asm volatile("v_mfma_f32_16x16x128_f8f6f4 %0, %1, %2, %0 cbsz:1 blgp:1" : "+v"(acc) : "a"(w), "v"(x));
}
__device__ __forceinline__ unsigned ld_sys(const void* p) {
  unsigned v;
  asm volatile("global_load_dword %0, %1, off sc0 sc1\n\ts_waitcnt vmcnt(0)" : "=&v"(v) : "v"(p) : "memory");
  return v;
}
__device__ __forceinline__ void st_sys(void* p, unsigned v) {
  asm volatile("global_store_dword %0, %1, off sc0 sc1" :: "v"(p), "v"(v) : "memory");
}
__device__ __forceinline__ bool spin_ge(const void* p, unsigned target, unsigned& f, char* wsl, unsigned call) {
  unsigned long long t0 = __builtin_amdgcn_s_memrealtime();
  for (unsigned n = 0;; ++n) {
    f = ld_sys(p);
    if (__builtin_amdgcn_ballot_w64(f < target) == 0) return true;
    if ((n & 63) == 63) {
      unsigned t = ld_sys(wsl + OFF_TMO);
      if (__builtin_amdgcn_readfirstlane(t) == call || __builtin_amdgcn_s_memrealtime() - t0 > DEADLINE) {
        st_sys(wsl + OFF_TMO, call);
        return false;
      }
    }
    __builtin_amdgcn_s_sleep(1);
  }
}

__device__ __forceinline__ void st16_sys(void* p, const v4u& v) {
  asm volatile("global_store_dwordx4 %0, %1, off sc0 sc1\n\ts_nop 1" :: "v"(p), "v"(v) : "memory");
}

constexpr unsigned REP = 0x01010101u;

__global__ void __launch_bounds__(256, 1) fused_ag_gemm(Params P) {
  extern __shared__ __attribute__((aligned(16))) char smem[];
  const int b = blockIdx.x, xcd = b & 7, slot = b >> 3;
  const int wave = __builtin_amdgcn_readfirstlane(threadIdx.x >> 6), lane = threadIdx.x & 63;
  const int g = lane >> 4, m = lane & 15;
  const unsigned rank = (unsigned)P.rank & 7u;
  char* const wsl = P.wsl;
  char* const wsp = P.ws[xcd];
  float sx = P.sx[0], sw = P.sw[0];
  STAMP(0);
  asm volatile("" :: "s"(P.x), "s"(P.w), "s"(P.out), "s"(wsl), "s"(wsp), "s"(rank));

#define GMAP 0
  char* const cntp = wsl + OFF_CNT + b * 128 + wave * 4;
  const unsigned lds0 = (unsigned)(size_t)(LAS char*)smem;
  __amdgpu_buffer_rsrc_t rw = __builtin_amdgcn_make_buffer_rsrc((void*)P.w, 0, K * NFULL, 0x00020000);
  constexpr int NT = GMAP == 0 ? 1 : 2;
  auto touch_w = [&](int sl) {
    const unsigned id = sl * 256 + threadIdx.x;
    const unsigned col = GMAP == 0 ? xcd * 256 + (id & 1) * 128 : (xcd >> 1) * 512 + (id & 3) * 128;
    const unsigned row = GMAP == 0 ? id >> 1 : id >> 2;
    #pragma unroll
    for (int q = 0; q < NT; ++q)
      asm volatile("s_mov_b32 m0, %0\n\ts_nop 0\n\tbuffer_load_dword %1, %2, %3 offen lds"
                   :: "s"(lds0 + 131072 + wave * 256), "v"(row * NFULL + rank * NS + col), "s"(rw), "s"((unsigned)(q * 2048 * NFULL)) : "memory");
  };
  auto touch = [&](const void* p) {
    asm volatile("s_mov_b32 m0, %0\n\ts_nop 0\n\tglobal_load_lds_dword %1, off sc0 sc1"
                 :: "s"(lds0 + 131072 + wave * 256), "v"(p) : "memory");
  };
  unsigned call;

  if (slot < 4) {
    const int q = slot;
    __builtin_amdgcn_s_waitcnt(0x0F70);
    if (xcd == NRG) {
      if (wave != 0) return;
      char* peer = P.ws[0];
      #pragma unroll
      for (int r = 1; r < NRG; ++r) peer = lane == r ? P.ws[r] : peer;
      const int l7 = lane < NRG ? lane : 0;
      call = __builtin_amdgcn_readfirstlane(ld_sys(cntp)) + 1;
      touch(peer + OFF_PAD + (32 + (6 - l7) * 4 + q) * 128);
      if (lane == 0) st_sys(cntp, call);
      unsigned f;
      if (!spin_ge(wsl + OFF_FLX + ((lane & 31) * 4 + q) * 128, call, f, wsl, call)) return;
      STAMP(2);
      if (lane < NRG) st_sys(peer + OFF_FGO + ((6 - lane) * 4 + q) * 128, call);
      STAMP_OUT(0);
      return;
    }
    const int rg = xcd, rgp = 6 - xcd;
    bool ok = true;
    if (wave < NPUSH) {
      v4u X[16]; unsigned c;
      const char* xs = P.x + (size_t)(wave * 16 + m) * K + q * 1024 + g * 16;
      asm volatile("global_load_dword %0, %1, off sc0 sc1" : "=&v"(c) : "v"(cntp) : "memory");
      #pragma unroll
      for (int i = 0; i < 16; ++i) asm volatile("global_load_dwordx4 %0, %1, off offset:%2" : "=&v"(X[i]) : "v"(xs), "i"(i * 64) : "memory");
      asm volatile("s_waitcnt vmcnt(15)" : "+v"(c), "+v"(X[0]) :: "memory");
      call = __builtin_amdgcn_readfirstlane(c) + 1;
      char* d = wsp + OFF_PSH + (call & 1) * PSH_PAR + (rgp * 4 + wave) * PIECE + q * 16384;
      #pragma unroll
      for (int i = 0; i < 16; ++i)
        asm volatile("s_waitcnt vmcnt(15)\n\tglobal_store_dwordx4 %0, %1, %2 offset:%3 sc0 sc1\n\ts_nop 1"
                     :: "v"(lane * 16), "v"(X[i]), "s"(d + (i >> 2) * 4096), "i"((i & 3) * 1024) : "memory");
      STAMP(1);
      asm volatile("s_waitcnt vmcnt(%0)" :: "i"(KDRAIN) : "memory");
      if (lane == 0) st_sys(cntp, call);
    } else {
      if (lane < 2) touch(wsp + OFF_PK + lane * PK_PAR + PK_PAD + (rgp * 4 + q) * 128);
      call = __builtin_amdgcn_readfirstlane(ld_sys(cntp)) + 1;
      if (lane == 0) st_sys(cntp, call);
      if (wave == 3) {
        unsigned f;
        ok = spin_ge(wsl + OFF_FGO + (rg * 4 + q) * 128, call, f, wsl, call);
        *(LAS volatile unsigned*)(smem + 65536) = ok ? 1u : 0u;
        asm volatile("s_waitcnt lgkmcnt(0)" ::: "memory");
      }
    }
    __builtin_amdgcn_s_barrier();
    asm volatile("" ::: "memory");
    STAMP(2);

    const int v = q * 4 + wave;
    const char* pk = wsp + OFF_PK + (call & 1) * PK_PAR;
    __amdgpu_buffer_rsrc_t rpk = __builtin_amdgcn_make_buffer_rsrc((void*)pk, 0, (int)PK_LS, 0x00020000);
    __amdgpu_buffer_rsrc_t rpl = __builtin_amdgcn_make_buffer_rsrc((void*)(wsp + OFF_PL + (call & 1) * PL_PAR), 0, (int)PL_PAR, 0x00020000);
    char* rdst = wsl + OFF_DST + rg * SHARD + v * 4096;
    const unsigned floff = OFF_FLP + (q * 32 + wave * 8 + (lane & 7)) * 128;
    char* flag = wsl + floff + rg * 4 * 32 * 128;
    LAS char* ll = (LAS char*)smem + wave * 8192;
    const unsigned l16 = lane * 16;
    if (__builtin_amdgcn_readfirstlane(*(LAS volatile unsigned*)(smem + 65536)) == 0) return;
    v4u A[4], B[4];
    #define LDM(R, j) asm volatile("buffer_load_dwordx4 %0, %3, %4, %5 offen sc0 sc1\n\t" \
                                   "buffer_load_dwordx4 %1, %3, %4, %5 offen offset:1024 sc0 sc1\n\t" \
                                   "buffer_load_dwordx4 %2, %3, %4, %5 offen offset:2048 sc0 sc1" \
                                   : "=&v"(R[0]), "=&v"(R[1]), "=&v"(R[2]) : "v"(l16), "s"(rpk), "s"((unsigned)((j) * 16 + v) * PKU) : "memory")
    #define LDP(R, j) asm volatile("s_nop 4\n\tbuffer_load_dwordx4 %0, %1, %2, %3 offen sc0 sc1" : "=&v"(R[3]) : "v"(l16), "s"(rpl), \
                                   "s"(__builtin_amdgcn_readlane(plo, (j))) : "memory")
    #pragma unroll
    for (int i = 0; i < 4; ++i)
      asm volatile("s_mov_b32 m0, %0\n\ts_nop 0\n\tglobal_load_lds_dwordx4 %1, off sc0 sc1"
                   :: "s"(lds0 + wave * 8192 + i * 1024), "v"(pk + PK_LS + v * 4096 + i * 1024 + l16) : "memory");
    LDM(A, NPUSH);
    asm volatile("s_waitcnt vmcnt(3)" ::: "memory");
    unsigned hdr = *(LAS volatile unsigned*)(ll + (lane & 31) * 128);
    const unsigned plo = (lane < 32 && (hdr & 256u)) ? (unsigned)(lane * 16 + v) * 1024u : PL_PAR;
    hdr = (hdr & 127u) * REP;
    LDP(A, NPUSH);
    LDM(B, NPUSH + 1);
    LDP(B, NPUSH + 1);
    auto plane = [&](int j) -> v4u {
      *(LAS volatile v4u*)(ll + 4096 + l16) = v4u{0u, 0u, 0u, 0u};
      unsigned e = *(LAS volatile unsigned*)(ll + (j & 31) * 128 + (lane & 31) * 4);
      e = (lane >= 1 && lane < 32) ? e : 0u;
      __hip_atomic_fetch_or((LAS unsigned*)(ll + 4096 + ((e >> 4) & 0x3fcu)), (e & 3u) << ((e >> 1) & 30u), __ATOMIC_RELAXED, __HIP_MEMORY_SCOPE_WAVEFRONT);
      e >>= 16;
      __hip_atomic_fetch_or((LAS unsigned*)(ll + 4096 + ((e >> 4) & 0x3fcu)), (e & 3u) << ((e >> 1) & 30u), __ATOMIC_RELAXED, __HIP_MEMORY_SCOPE_WAVEFRONT);
      return *(LAS volatile v4u*)(ll + 4096 + l16);
    };
    const unsigned base = call * 64;
    auto iter = [&](v4u (&R)[4], int j) {
      v4u pl = plane(j);
      const unsigned Bb = __builtin_amdgcn_readlane(hdr, j);
      if (j == NPUSH) asm volatile("s_waitcnt vmcnt(4)" : "+v"(R[0]), "+v"(R[1]), "+v"(R[2]), "+v"(R[3]), "+v"(pl) :: "memory");
      else if (j == NPUSH + 1) asm volatile("s_waitcnt vmcnt(9)" : "+v"(R[0]), "+v"(R[1]), "+v"(R[2]), "+v"(R[3]), "+v"(pl) :: "memory");
      else asm volatile("s_waitcnt vmcnt(10)" : "+v"(R[0]), "+v"(R[1]), "+v"(R[2]), "+v"(R[3]), "+v"(pl) :: "memory");
      v4u Y[4];
      #pragma unroll
      for (int e = 0; e < 4; ++e) {
        const unsigned q0 = R[0][e], q1 = R[1][e], q2 = R[2][e], p = pl[e] | R[3][e];
        const unsigned p3 = (__builtin_amdgcn_alignbit(q0, q0, 6) & (0x81u * REP)) | ((q1 >> 4) & (0x06u * REP)) | ((q2 >> 2) & (0x18u * REP));
        Y[0][e] = ((q0 & (0x9fu * REP)) + Bb) ^ ((p << 5) & (0x60u * REP));
        Y[1][e] = ((q1 & (0x9fu * REP)) + Bb) ^ ((p << 3) & (0x60u * REP));
        Y[2][e] = ((q2 & (0x9fu * REP)) + Bb) ^ ((p << 1) & (0x60u * REP));
        Y[3][e] = (p3 + Bb) ^ ((p >> 1) & (0x60u * REP));
      }
      asm volatile("global_store_dwordx4 %4, %0, %5 sc0 sc1\n\t"
                   "global_store_dwordx4 %4, %1, %5 offset:1024 sc0 sc1\n\t"
                   "global_store_dwordx4 %4, %2, %5 offset:2048 sc0 sc1\n\t"
                   "global_store_dwordx4 %4, %3, %5 offset:3072 sc0 sc1\n\ts_nop 1"
                   :: "v"(Y[0]), "v"(Y[1]), "v"(Y[2]), "v"(Y[3]), "v"(l16), "s"(rdst + (unsigned)j * PIECE) : "memory");
      LDM(R, j + 2);
      LDP(R, j + 2);
      asm volatile("s_waitcnt vmcnt(13)" ::: "memory");
      __builtin_amdgcn_s_barrier();
      if (lane < 8) {
        if (NPUSH > 0 && j == NPUSH) st_sys(wsp + floff + rgp * 4 * 32 * 128 + 4, base);
        else st_sys(flag, base + (unsigned)(j - NPUSH));
      }
      STAMP(8 + j);
    };
    iter(A, NPUSH);
    iter(B, NPUSH + 1);
    int j = NPUSH + 2;
    for (; j + 1 < NPIECE; j += 2) { iter(A, j); iter(B, j + 1); }
    if (j < NPIECE) iter(A, j);
    asm volatile("s_waitcnt vmcnt(1)" : "+v"(A[0]), "+v"(A[1]), "+v"(A[2]), "+v"(A[3]), "+v"(B[0]), "+v"(B[1]), "+v"(B[2]), "+v"(B[3]) :: "memory");
    __builtin_amdgcn_s_barrier();
    if (lane < 8) st_sys(flag, base + (unsigned)(NPIECE - NPUSH));
    STAMP(8 + NPIECE);
    STAMP(3);
    STAMP_OUT(0);
    return;
  }

  const int i = slot - 4;
  int rg, cb;
  if (GMAP == 0) { rg = i >> 2; cb = xcd * 4 + (i & 3); }
  else {
    const int r = i >> 3, o = xcd & 1;
    rg = r < 2 ? 2 * o + r : (r == 2 ? 4 + o : 6); cb = (xcd >> 1) * 8 + (r < 3 ? (i & 7) : o * 4 + (i & 3));
  }
  const bool stager = rg < 4;
  if (stager) {
    const int task = rg * 32 + cb;
    const int unit = task * 4 + wave;
    unsigned c; v4u X[4];
    const char* xs = P.x + (size_t)((task >> 2) * 16 + m) * K + (unit & 15) * 256 + g * 16;
    asm volatile("global_load_dwordx4 %0, %5, off\n\t"
                 "global_load_dwordx4 %1, %5, off offset:64\n\t"
                 "global_load_dwordx4 %2, %5, off offset:128\n\t"
                 "global_load_dwordx4 %3, %5, off offset:192\n\t"
                 "global_load_dword %4, %6, off sc0 sc1"
                 : "=&v"(X[0]), "=&v"(X[1]), "=&v"(X[2]), "=&v"(X[3]), "=&v"(c) : "v"(xs), "v"(cntp) : "memory");
    touch_w(slot);
    LAS char* ll = (LAS char*)smem + wave * 32768;
    *(LAS volatile unsigned*)(ll + (lane & 31) * 4) = 0u;
    asm volatile("s_waitcnt vmcnt(%5)" : "+v"(c), "+v"(X[0]), "+v"(X[1]), "+v"(X[2]), "+v"(X[3]) : "i"(NT) : "memory");
    STAMP(6);
    call = __builtin_amdgcn_readfirstlane(c) + 1;
    unsigned sum = 0;
    #pragma unroll
    for (int e = 0; e < 4; ++e) sum = __builtin_amdgcn_sad_u8(X[0][e] & (0x7fu * REP), 0u, sum);
    sum += __builtin_amdgcn_update_dpp(0u, sum, 0x111, 0xf, 0xf, true);
    sum += __builtin_amdgcn_update_dpp(0u, sum, 0x112, 0xf, 0xf, true);
    sum += __builtin_amdgcn_update_dpp(0u, sum, 0x114, 0xf, 0xf, true);
    sum += __builtin_amdgcn_update_dpp(0u, sum, 0x118, 0xf, 0xf, true);
    int bs = (__builtin_amdgcn_readlane(sum, 15) + __builtin_amdgcn_readlane(sum, 31) + __builtin_amdgcn_readlane(sum, 47) + __builtin_amdgcn_readlane(sum, 63) - 20838) >> 10;
    bs = bs < 0 ? 0 : (bs > 96 ? 96 : bs);
    const unsigned Bb = (unsigned)bs * REP;
    v4u Q[3], PLq;
    #pragma unroll
    for (int e = 0; e < 4; ++e) {
      unsigned p[4], mk[4];
      #pragma unroll
      for (int i = 0; i < 4; ++i) {
        const unsigned x = X[i][e];
        p[i] = (((x | (0x80u * REP)) - Bb) & (0x1fu * REP)) | (x & (0x80u * REP));
        mk[i] = x ^ (p[i] + Bb);
      }
      Q[0][e] = p[0] | (__builtin_amdgcn_alignbit(p[3], p[3], 26) & (0x60u * REP));
      Q[1][e] = p[1] | ((p[3] << 4) & (0x60u * REP));
      Q[2][e] = p[2] | ((p[3] << 2) & (0x60u * REP));
      PLq[e] = (mk[0] >> 5) | (mk[1] >> 3) | (mk[2] >> 1) | (mk[3] << 1);
    }
    char* pk = wsl + OFF_PK + (call & 1) * PK_PAR;
    #pragma unroll
    for (int i = 0; i < 3; ++i) st16_sys(pk + unit * PKU + i * 1024 + lane * 16, Q[i]);
    #pragma unroll
    for (int i = 0; i < 4; ++i) st16_sys(wsl + OFF_RAW + unit * 4096 + i * 1024 + lane * 16, X[i]);
    int at = 0;
    #pragma unroll
    for (int e = 0; e < 4; ++e) at += __builtin_popcount((PLq[e] | (PLq[e] >> 1)) & 0x55555555u);
    int inc = at;
    inc += __builtin_amdgcn_update_dpp(0, inc, 0x111, 0xf, 0xf, true);
    inc += __builtin_amdgcn_update_dpp(0, inc, 0x112, 0xf, 0xf, true);
    inc += __builtin_amdgcn_update_dpp(0, inc, 0x114, 0xf, 0xf, true);
    inc += __builtin_amdgcn_update_dpp(0, inc, 0x118, 0xf, 0xf, true);
    inc += __builtin_amdgcn_update_dpp(0, inc, 0x142, 0xa, 0xf, false);
    inc += __builtin_amdgcn_update_dpp(0, inc, 0x143, 0xc, 0xf, false);
    const bool ovf = __builtin_amdgcn_readlane(inc, 63) > LCAP;
    if (!ovf) {
      LAS char* lp = ll + 4 + (inc - at) * 2;
      #pragma unroll
      for (int e = 0; e < 4; ++e) {
        unsigned t = PLq[e];
        while (t != 0) {
          const unsigned pos = (unsigned)__builtin_ctz(t) & 30u;
          *(LAS volatile unsigned short*)lp = (unsigned short)(((lane * 4 + e) << 6) | (pos << 1) | ((t >> pos) & 3u));
          lp += 2;
          t &= ~(3u << pos);
        }
      }
    }
    if (ovf) st16_sys(wsl + OFF_PL + (call & 1) * PL_PAR + unit * 1024 + lane * 16, PLq);
    unsigned le = *(LAS volatile unsigned*)(ll + (lane & 31) * 4);
    le = lane == 0 ? (unsigned)bs | (ovf ? 256u : 0u) : le;
    if (lane < 32) st_sys(pk + PK_LS + ((unit & 15) * 32 + (unit >> 4)) * 128 + lane * 4, le);
    STAMP(7);
    asm volatile("s_waitcnt vmcnt(0)" ::: "memory");
    __builtin_amdgcn_s_barrier();
    if (threadIdx.x == 0) st_sys(wsl + OFF_FLX + task * 128, call);
  } else {
    call = 0;
    touch_w(slot);
    if (slot >= 28) touch_w(slot - 28);
  }
  STAMP(1);

  const int rot = (rg + cb) & 7;

  v8i Wf[8][4];
  {
    char* wl = smem + wave * 32768;
    const int r16 = lane >> 2;
    const unsigned vb = (unsigned)(wave * 1024 + 2 * r16) * NFULL + rank * NS + cb * 64;
    const unsigned v0 = vb + (((lane & 3) ^ ((r16 >> 3) << 1)) << 4), v1 = v0 ^ 16;
    auto issue = [&](int t, int q0, int q1) {
      #pragma unroll
      for (int q = q0; q < q1; ++q)
        asm volatile("s_mov_b32 m0, %0\n\ts_nop 0\n\tbuffer_load_dwordx4 %1, %2, %3 offen lds"
                     :: "s"(lds0 + wave * 32768 + (t & 3) * 8192 + q * 1024), "v"((q & 1) ? v1 : v0), "s"(rw),
                        "s"((unsigned)((((t + rot) & 7) * 128 + (q & 1) + (q >> 1) * 32) * NFULL)) : "memory");
    };
    issue(0, 0, 8); issue(1, 0, 8); issue(2, 0, 8); issue(3, 0, 8);
    const int q8 = (lane & 15) >> 1, p = lane & 1;
    #pragma unroll
    for (int t = 0; t < 8; ++t) {
      if (t <= 4) asm volatile("s_waitcnt vmcnt(24)" ::: "memory");
      else if (t == 5) asm volatile("s_waitcnt vmcnt(16)" ::: "memory");
      else if (t == 6) asm volatile("s_waitcnt vmcnt(8)" ::: "memory");
      else asm volatile("s_waitcnt vmcnt(0)" ::: "memory");
      #pragma unroll
      for (int c = 0; c < 4; ++c) {
        v2i r[4];
        #pragma unroll
        for (int he = 0; he < 4; ++he) {
          const int row = 64 * (he >> 1) + 16 * g + 8 * (he & 1) + q8;
          const int lrow = 16 * ((row & 1) + 2 * (row >> 5)) + ((row & 31) >> 1);
          const int f = (row & 1) | (((row >> 4) & 1) << 1);
          r[he] = __builtin_amdgcn_ds_read_tr8_b64_v2i32((LAS v2i*)(wl + (t & 3) * 8192 + lrow * 64 + ((c ^ f) << 4) + p * 8));
        }
        v8i w = {r[0].x, r[0].y, r[1].x, r[1].y, r[2].x, r[2].y, r[3].x, r[3].y};
        Wf[t][c] = w;
      }
      asm volatile("s_waitcnt lgkmcnt(0)" : "+a"(Wf[t][0]), "+a"(Wf[t][1]), "+a"(Wf[t][2]), "+a"(Wf[t][3]) :: "memory");
      if (t < 4) issue(t + 4, 0, 8);
    }
  }
  STAMP(2);
  if (!stager) call = __builtin_amdgcn_readfirstlane(ld_sys(cntp)) + 1;
  if (lane == 0) st_sys(cntp, call);

  float scale = sx * sw;
#define OWNS 4
  constexpr int REST = NPIECE - 4 * OWNS;
  const int own0 = stager ? rg * OWNS : 4 * OWNS + (REST * (rg - 4)) / 3;
  const int nown = stager ? OWNS : 4 * OWNS + (REST * (rg - 3)) / 3 - own0;
  const int np = nown + NPIECE;
  const unsigned src = (rank + 1 + rg) & 7u;
  bool alive = true;
  {
    unsigned f;
    const int jj = own0 + ((lane & 15) < nown ? (lane & 15) : 0);
    if (!spin_ge(wsl + OFF_FLX + (jj * 4 + wave) * 128, call, f, wsl, call)) { alive = false; scale = __builtin_nanf(""); }
  }
  const int rj = lane - nown;
  const unsigned t_x = lane < nown ? OFF_RAW + (unsigned)(own0 + lane) * PIECE
                     : rj < NPUSH ? OFF_PSH + (call & 1) * PSH_PAR + (unsigned)(rg * 4 + rj) * PIECE
                     : rj < NPIECE ? OFF_DST + rg * SHARD + (unsigned)rj * PIECE : OFF_END;
  const unsigned t_o = (lane < nown ? rank * MS + (unsigned)(own0 + lane) * 16 : src * MS + (unsigned)rj * 16) * (NS * 4);
  const unsigned t_f = (rj >= 0 && rj < NPIECE) ? call * 64 + (unsigned)(rj < NPUSH ? 0 : rj + 1 - NPUSH) : 0u;
  const unsigned t_w = (rj >= 0 && rj < NPUSH) ? 1u : 0u;
  auto xoff = [&](int n) -> unsigned { return __builtin_amdgcn_readlane(t_x, n); };
  __amdgpu_buffer_rsrc_t rs = __builtin_amdgcn_make_buffer_rsrc((void*)wsl, 0, (int)OFF_END, 0x00020000);
  __amdgpu_buffer_rsrc_t ro = __builtin_amdgcn_make_buffer_rsrc((void*)P.out, 0, M * NS * 4, 0x00020000);
  constexpr int AUX = 17;
  const unsigned voff = wave * 16384 + lane * 16;
  const char* const flagp = wsl + OFF_FLP + ((rg * 4 + wave) * 32 + cb) * 128;
  v4u xb[16];
  unsigned sb = xoff(0);
  {
    #pragma unroll
    for (int t = 0; t < 16; ++t) xb[t] = __builtin_amdgcn_raw_buffer_load_b128(rs, voff + (t & 1) * 1024, sb + ((((t >> 1) + rot) & 7) << 11), AUX);
  }
  __builtin_amdgcn_s_barrier();

  const unsigned wr_off = wave * 4096 + m * 256;
  const int rrow = 4 * wave + g;
  const unsigned rd_off = rrow * 256 + ((m ^ rrow) << 4);
  const unsigned ooff = (rrow * NS + cb * 64 + m * 4) * 4;
  v4f acc[2][4];
#define PGAP 13
  unsigned fl = 0;
  __amdgpu_buffer_rsrc_t rf = __builtin_amdgcn_make_buffer_rsrc((void*)flagp, 0, 128, 0x00020000);
  auto step = [&](auto Bc, int n, bool prev, bool cur) {
    constexpr int B = decltype(Bc)::value;
    const unsigned sbp = sb;
    char* rb = smem + (B ^ 1) * 16384;
    v4f s0, s1, s2, s3, s;
    #pragma unroll
    for (int t = 0; t < 8; ++t) {
      const v4u lo = xb[2 * t], hi = xb[2 * t + 1];
      const v8i x = {(int)lo.x, (int)lo.y, (int)lo.z, (int)lo.w, (int)hi.x, (int)hi.y, (int)hi.z, (int)hi.w};
      #pragma unroll
      for (int c = 0; c < 4; ++c) {
        const int gap = t * 4 + c;
        if (cur) { if (t == 0) mfma0(acc[B][c], Wf[t][c], x); else mfma(acc[B][c], Wf[t][c], x); }
        SB;
        if (cur && gap == 3) {
          const unsigned target = __builtin_amdgcn_readlane(t_f, n + 1);
          sb = xoff(n + 1);
          const unsigned word = __builtin_amdgcn_readlane(t_w, n + 1);
          if ((unsigned)__builtin_amdgcn_readlane(fl, word) < target && alive) {
            unsigned f;
            if (!spin_ge(flagp + word * 4, target, f, wsl, call)) { alive = false; scale = __builtin_nanf(""); }
            STAMP(8 + n + 2 - nown);
          }
        }
        if (cur && (c == 0 || c == 2)) {
          if (t > 0) xb[2 * (t - 1) + (c >> 1)] = __builtin_amdgcn_raw_buffer_load_b128(rs, voff + (c >> 1) * 1024, sb + (((t - 1 + rot) & 7) << 11), AUX);
          else if (prev) xb[14 + (c >> 1)] = __builtin_amdgcn_raw_buffer_load_b128(rs, voff + (c >> 1) * 1024, sbp + (((7 + rot) & 7) << 11), AUX);
        }
        if (cur && gap == PGAP) fl = __builtin_amdgcn_raw_buffer_load_b32(rf, (lane & 1) * 4, 0, AUX);
        if (prev) {
          if (gap == 0 && !cur) asm volatile("s_nop 7\n\ts_nop 7" ::: "memory");
          if (gap == 5 || gap == 7 || gap == 9 || gap == 11) { const int cc = (gap - 5) >> 1; *(LAS v4f*)(rb + wr_off + (((4 * cc + g) ^ m) << 4)) = acc[B ^ 1][cc]; }
          if (gap == 15) { asm volatile("s_waitcnt lgkmcnt(0)" ::: "memory"); __builtin_amdgcn_s_barrier(); asm volatile("" ::: "memory"); }
          if (gap == 17) { s0 = *(LAS v4f*)(rb + rd_off); s1 = *(LAS v4f*)(rb + 4096 + rd_off); }
          if (gap == 19) { s2 = *(LAS v4f*)(rb + 8192 + rd_off); s3 = *(LAS v4f*)(rb + 12288 + rd_off); }
          if (gap == 23) { for (int e = 0; e < 4; ++e) asm("v_add_f32 %0, %1, %2" : "=v"(s[e]) : "v"(s0[e]), "v"(s1[e])); }
          if (gap == 25) { for (int e = 0; e < 4; ++e) asm("v_add_f32 %0, %1, %2" : "=v"(s2[e]) : "v"(s2[e]), "v"(s3[e])); }
          if (gap == 27) { for (int e = 0; e < 4; ++e) asm("v_add_f32 %0, %1, %2" : "=v"(s[e]) : "v"(s[e]), "v"(s2[e])); }
          if (gap == 29) { for (int e = 0; e < 4; ++e) asm("v_mul_f32 %0, %1, %2" : "=v"(s[e]) : "v"(scale), "v"(s[e])); }
          if (gap == 31) __builtin_amdgcn_raw_buffer_store_b128(__builtin_bit_cast(v4u, s), ro, ooff, __builtin_amdgcn_readlane(t_o, n - 1), 0);
        }
        SB;
      }
    }
  };
  std::integral_constant<int, 0> I0; std::integral_constant<int, 1> I1;
  step(I0, 0, false, true);
  STAMP(4);
  int n = 1;
  for (; n + 1 < np; n += 2) {
    step(I1, n, true, true); step(I0, n + 1, true, true);
    STAMP(44 + (n >> 1));
  }
  if (n < np) { step(I1, n, true, true); step(I0, n + 1, true, false); }
  else step(I1, n, true, false);
  STAMP(3);
  STAMP_OUT(0);
}

}

extern "C" void kernel_launch(int rank, int nranks, ncclComm_t comm, hipStream_t stream,
                              void* const* d_in, const int* in_sizes, int n_in,
                              void* d_out, int out_size,
                              const int* in_dtypes, int out_dtype,
                              void* const* d_out_all,
                              void* const* d_ws_all, size_t ws_bytes) {
  (void)comm; (void)in_dtypes; (void)out_dtype; (void)d_out_all;
  if (nranks != NR || n_in != 4 || (unsigned)rank >= (unsigned)NR || ws_bytes < WS_USED ||
      in_sizes[0] != MS * K || in_sizes[1] != K * NFULL || out_size != M * NS) {
    fprintf(stderr, "kernel_launch: unexpected problem shape\n");
    return;
  }
  HIP_CHECK(hipFuncSetAttribute((const void*)fused_ag_gemm, hipFuncAttributeMaxDynamicSharedMemorySize, LDS_BYTES));
  Params P;
  for (int r = 0; r < NR; ++r) P.ws[r] = (char*)d_ws_all[(rank + 1 + r) % NR];
  P.wsl = (char*)d_ws_all[rank];
  P.x = (const char*)d_in[0];
  P.w = (const char*)d_in[1];
  P.sx = (const float*)d_in[2];
  P.sw = (const float*)d_in[3];
  P.out = (float*)d_out;
  P.rank = rank;
  P.pad = 0;
  fused_ag_gemm<<<NWG, 256, LDS_BYTES, stream>>>(P);
  HIP_CHECK(hipGetLastError());
}
```

```cpp
#include <hip/hip_runtime.h>
#include <rccl/rccl.h>
#include <cstdio>
#include <cstdint>
#include <type_traits>
#include <utility>

#define HIP_CHECK(expr) do { hipError_t _e = (expr); if (_e != hipSuccess) { \
    fprintf(stderr, "HIP error %s at %s:%d\n", hipGetErrorString(_e), __FILE__, __LINE__); \
    return; } } while (0)

typedef unsigned v4u __attribute__((ext_vector_type(4)));
typedef int v8i __attribute__((ext_vector_type(8)));
typedef int v2i __attribute__((ext_vector_type(2)));
typedef float v4f __attribute__((ext_vector_type(4)));
#define LAS __attribute__((address_space(3)))
#define SB __builtin_amdgcn_sched_barrier(0)

namespace {

constexpr int NR = 8;
constexpr int M = 4096, K = 4096, NFULL = 16384, NS = 2048, MS = 512;
constexpr unsigned SHARD = MS * K;
constexpr unsigned PIECE = 16 * K;
constexpr int NPIECE = MS / 16;
constexpr int NRG = 7;
constexpr int NWG = 256;
constexpr int LDS_BYTES = 163840;

#define NPUSH 3
#define KDRAIN 6
constexpr unsigned PKU = 3072;
constexpr int LCAP = 62;

constexpr unsigned OFF_RAW = 0;
constexpr unsigned OFF_DST = SHARD;
constexpr unsigned OFF_PSH = 8 * SHARD;
constexpr unsigned PSH_PAR = SHARD;
constexpr unsigned OFF_END = 10 * SHARD;
constexpr unsigned OFF_PK  = OFF_END;
constexpr unsigned PK_PAR = SHARD, PK_LS = 512 * PKU, PK_PAD = PK_LS + 512 * 128;
constexpr unsigned OFF_PL  = 12 * SHARD;
constexpr unsigned PL_PAR = 512 * 1024;
constexpr unsigned OFF_FLX = 13 * SHARD;
constexpr unsigned OFF_FGO = OFF_FLX + 128 * 128;
constexpr unsigned OFF_FLP = OFF_FGO + NRG * 4 * 128;
constexpr unsigned OFF_CNT = OFF_FLP + NRG * 4 * 32 * 128;
constexpr unsigned OFF_TMO = OFF_CNT + NWG * 128;
constexpr unsigned OFF_PAD = OFF_TMO + 128;
constexpr unsigned OFF_DBG = OFF_PAD + 64 * 128;
constexpr unsigned OFF_DB2 = OFF_DBG + NWG * 64 * 4;
constexpr unsigned WS_USED = OFF_DB2 + 4096 * 64 * 4;
constexpr unsigned long long DEADLINE = 30000000ull;
static_assert(NPUSH >= 0 && NPUSH <= 3 && 7 * 4 * PIECE <= PSH_PAR && PK_PAD + 64 * 128 <= PK_PAR, "layout");

struct Params {
  char* ws[NR];
  char* wsl;
  const char* x;
  const char* w;
  const float* sx;
  const float* sw;
  float* out;
  int rank;
  int pad;
};

#define STAMP(i) do {} while (0)
#define STAMP_OUT(w) do {} while (0)

__device__ __forceinline__ void mfma0(v4f& acc, const v8i& w, const v8i& x) {
  asm volatile("v_mfma_f32_16x16x128_f8f6f4 %0, %1, %2, 0 cbsz:1 blgp:1" : "=&v"(acc) : "a"(w), "v"(x));
}
__device__ __forceinline__ void mfma(v4f& acc, const v8i& w, const v8i& x) {
  asm volatile("v_mfma_f32_16x16x128_f8f6f4 %0, %1, %2, %0 cbsz:1 blgp:1" : "+v"(acc) : "a"(w), "v"(x));
}
__device__ __forceinline__ void mfma0v(v4f& acc, const v8i& w, const v8i& x) {
  asm volatile("v_mfma_f32_16x16x128_f8f6f4 %0, %1, %2, 0 cbsz:1 blgp:1" : "=&v"(acc) : "v"(w), "v"(x));
}
__device__ __forceinline__ void mfmav(v4f& acc, const v8i& w, const v8i& x) {
  asm volatile("v_mfma_f32_16x16x128_f8f6f4 %0, %1, %2, %0 cbsz:1 blgp:1" : "+v"(acc) : "v"(w), "v"(x));
}
__device__ __forceinline__ v8i cat(const v4u& lo, const v4u& hi) {
  return v8i{(int)lo.x, (int)lo.y, (int)lo.z, (int)lo.w, (int)hi.x, (int)hi.y, (int)hi.z, (int)hi.w};
}
template <class F, int... Is> __device__ __forceinline__ void sfor_impl(F&& f, std::integer_sequence<int, Is...>) { (f(std::integral_constant<int, Is>{}), ...); }
template <int N, class F> __device__ __forceinline__ void sfor(F&& f) { sfor_impl(f, std::make_integer_sequence<int, N>{}); }
__device__ __forceinline__ unsigned ld_sys(const void* p) {
  unsigned v;
  asm volatile("global_load_dword %0, %1, off sc0 sc1\n\ts_waitcnt vmcnt(0)" : "=&v"(v) : "v"(p) : "memory");
  return v;
}
__device__ __forceinline__ void st_sys(void* p, unsigned v) {
  asm volatile("global_store_dword %0, %1, off sc0 sc1" :: "v"(p), "v"(v) : "memory");
}
__device__ __forceinline__ bool spin_ge(const void* p, unsigned target, unsigned& f, char* wsl, unsigned call) {
  unsigned long long t0 = __builtin_amdgcn_s_memrealtime();
  for (unsigned n = 0;; ++n) {
    f = ld_sys(p);
    if (__builtin_amdgcn_ballot_w64(f < target) == 0) return true;
    if ((n & 63) == 63) {
      unsigned t = ld_sys(wsl + OFF_TMO);
      if (__builtin_amdgcn_readfirstlane(t) == call || __builtin_amdgcn_s_memrealtime() - t0 > DEADLINE) {
        st_sys(wsl + OFF_TMO, call);
        return false;
      }
    }
    __builtin_amdgcn_s_sleep(1);
  }
}

__device__ __forceinline__ void st16_sys(void* p, const v4u& v) {
  asm volatile("global_store_dwordx4 %0, %1, off sc0 sc1\n\ts_nop 1" :: "v"(p), "v"(v) : "memory");
}

constexpr unsigned REP = 0x01010101u;

__global__ void __launch_bounds__(256, 1) fused_ag_gemm(Params P) {
  extern __shared__ __attribute__((aligned(16))) char smem[];
  const int b = blockIdx.x, xcd = b & 7, slot = b >> 3;
  const int wave = __builtin_amdgcn_readfirstlane(threadIdx.x >> 6), lane = threadIdx.x & 63;
  const int g = lane >> 4, m = lane & 15;
  const unsigned rank = (unsigned)P.rank & 7u;
  char* const wsl = P.wsl;
  char* const wsp = P.ws[xcd];
  float sx = P.sx[0], sw = P.sw[0];
  STAMP(0);
  asm volatile("" :: "s"(P.x), "s"(P.w), "s"(P.out), "s"(wsl), "s"(wsp), "s"(rank));

#define GMAP 3
  char* const cntp = wsl + OFF_CNT + b * 128 + wave * 4;
  const unsigned lds0 = (unsigned)(size_t)(LAS char*)smem;
  __amdgpu_buffer_rsrc_t rw = __builtin_amdgcn_make_buffer_rsrc((void*)P.w, 0, K * NFULL, 0x00020000);
  constexpr int NT = GMAP == 0 ? 1 : 2;
  auto touch_w = [&](int sl) {
    const unsigned id = sl * 256 + threadIdx.x;
    const unsigned col = GMAP == 0 ? xcd * 256 + (id & 1) * 128 : (xcd >> 1) * 512 + (id & 3) * 128;
    const unsigned row = GMAP == 0 ? id >> 1 : id >> 2;
    #pragma unroll
    for (int q = 0; q < NT; ++q)
      asm volatile("s_mov_b32 m0, %0\n\ts_nop 0\n\tbuffer_load_dword %1, %2, %3 offen lds"
                   :: "s"(lds0 + wave * 40960), "v"(row * NFULL + rank * NS + col), "s"(rw), "s"((unsigned)(q * 2048 * NFULL)) : "memory");
  };
  auto touch = [&](const void* p) {
    asm volatile("s_mov_b32 m0, %0\n\ts_nop 0\n\tglobal_load_lds_dword %1, off sc0 sc1"
                 :: "s"(lds0 + wave * 40960), "v"(p) : "memory");
  };
  unsigned call;

  if (slot < 4) {
    const int q = slot;
    __builtin_amdgcn_s_waitcnt(0x0F70);
    if (xcd == NRG) {
      if (wave != 0) return;
      char* peer = P.ws[0];
      #pragma unroll
      for (int r = 1; r < NRG; ++r) peer = lane == r ? P.ws[r] : peer;
      const int l7 = lane < NRG ? lane : 0;
      call = __builtin_amdgcn_readfirstlane(ld_sys(cntp)) + 1;
      touch(peer + OFF_PAD + (32 + (6 - l7) * 4 + q) * 128);
      if (lane == 0) st_sys(cntp, call);
      unsigned f;
      if (!spin_ge(wsl + OFF_FLX + ((lane & 31) * 4 + q) * 128, call, f, wsl, call)) return;
      STAMP(2);
      if (lane < NRG) st_sys(peer + OFF_FGO + ((6 - lane) * 4 + q) * 128, call);
      STAMP_OUT(0);
      return;
    }
    const int rg = xcd, rgp = 6 - xcd;
    bool ok = true;
    if (wave < NPUSH) {
      v4u X[16]; unsigned c;
      const char* xs = P.x + (size_t)(wave * 16 + m) * K + q * 1024 + g * 16;
      asm volatile("global_load_dword %0, %1, off sc0 sc1" : "=&v"(c) : "v"(cntp) : "memory");
      #pragma unroll
      for (int i = 0; i < 16; ++i) asm volatile("global_load_dwordx4 %0, %1, off offset:%2" : "=&v"(X[i]) : "v"(xs), "i"(i * 64) : "memory");
      asm volatile("s_waitcnt vmcnt(15)" : "+v"(c), "+v"(X[0]) :: "memory");
      call = __builtin_amdgcn_readfirstlane(c) + 1;
      char* d = wsp + OFF_PSH + (call & 1) * PSH_PAR + (rgp * 4 + wave) * PIECE + q * 16384;
      #pragma unroll
      for (int i = 0; i < 16; ++i)
        asm volatile("s_waitcnt vmcnt(15)\n\tglobal_store_dwordx4 %0, %1, %2 offset:%3 sc0 sc1\n\ts_nop 1"
                     :: "v"(lane * 16), "v"(X[i]), "s"(d + (i >> 2) * 4096), "i"((i & 3) * 1024) : "memory");
      STAMP(1);
      asm volatile("s_waitcnt vmcnt(%0)" :: "i"(KDRAIN) : "memory");
      if (lane == 0) st_sys(cntp, call);
    } else {
      if (lane < 2) touch(wsp + OFF_PK + lane * PK_PAR + PK_PAD + (rgp * 4 + q) * 128);
      call = __builtin_amdgcn_readfirstlane(ld_sys(cntp)) + 1;
      if (lane == 0) st_sys(cntp, call);
      if (wave == 3) {
        unsigned f;
        ok = spin_ge(wsl + OFF_FGO + (rg * 4 + q) * 128, call, f, wsl, call);
        *(LAS volatile unsigned*)(smem + 4096) = ok ? 1u : 0u;
        asm volatile("s_waitcnt lgkmcnt(0)" ::: "memory");
      }
    }
    __builtin_amdgcn_s_barrier();
    asm volatile("" ::: "memory");
    STAMP(2);

    const int v = q * 4 + wave;
    const char* pk = wsp + OFF_PK + (call & 1) * PK_PAR;
    __amdgpu_buffer_rsrc_t rpk = __builtin_amdgcn_make_buffer_rsrc((void*)pk, 0, (int)PK_LS, 0x00020000);
    __amdgpu_buffer_rsrc_t rpl = __builtin_amdgcn_make_buffer_rsrc((void*)(wsp + OFF_PL + (call & 1) * PL_PAR), 0, (int)PL_PAR, 0x00020000);
    char* rdst = wsl + OFF_DST + rg * SHARD + v * 4096;
    const unsigned floff = OFF_FLP + (q * 32 + wave * 8 + (lane & 7)) * 128;
    char* flag = wsl + floff + rg * 4 * 32 * 128;
    LAS char* ll = (LAS char*)smem + wave * 40960 + 8192;
    const unsigned l16 = lane * 16;
    if (__builtin_amdgcn_readfirstlane(*(LAS volatile unsigned*)(smem + 4096)) == 0) return;
    v4u A[4], B[4];
    #define LDM(R, j) asm volatile("buffer_load_dwordx4 %0, %3, %4, %5 offen sc0 sc1\n\t" \
                                   "buffer_load_dwordx4 %1, %3, %4, %5 offen offset:1024 sc0 sc1\n\t" \
                                   "buffer_load_dwordx4 %2, %3, %4, %5 offen offset:2048 sc0 sc1" \
                                   : "=&v"(R[0]), "=&v"(R[1]), "=&v"(R[2]) : "v"(l16), "s"(rpk), "s"((unsigned)((j) * 16 + v) * PKU) : "memory")
    #define LDP(R, j) asm volatile("s_nop 4\n\tbuffer_load_dwordx4 %0, %1, %2, %3 offen sc0 sc1" : "=&v"(R[3]) : "v"(l16), "s"(rpl), \
                                   "s"(__builtin_amdgcn_readlane(plo, (j))) : "memory")
    #pragma unroll
    for (int i = 0; i < 4; ++i)
      asm volatile("s_mov_b32 m0, %0\n\ts_nop 0\n\tglobal_load_lds_dwordx4 %1, off sc0 sc1"
                   :: "s"(lds0 + wave * 40960 + 8192 + i * 1024), "v"(pk + PK_LS + v * 4096 + i * 1024 + l16) : "memory");
    LDM(A, NPUSH);
    asm volatile("s_waitcnt vmcnt(3)" ::: "memory");
    unsigned hdr = *(LAS volatile unsigned*)(ll + (lane & 31) * 128);
    const unsigned plo = (lane < 32 && (hdr & 256u)) ? (unsigned)(lane * 16 + v) * 1024u : PL_PAR;
    hdr = (hdr & 127u) * REP;
    LDP(A, NPUSH);
    LDM(B, NPUSH + 1);
    LDP(B, NPUSH + 1);
    auto plane = [&](int j) -> v4u {
      *(LAS volatile v4u*)(ll + 4096 + l16) = v4u{0u, 0u, 0u, 0u};
      unsigned e = *(LAS volatile unsigned*)(ll + (j & 31) * 128 + (lane & 31) * 4);
      e = (lane >= 1 && lane < 32) ? e : 0u;
      __hip_atomic_fetch_or((LAS unsigned*)(ll + 4096 + ((e >> 4) & 0x3fcu)), (e & 3u) << ((e >> 1) & 30u), __ATOMIC_RELAXED, __HIP_MEMORY_SCOPE_WAVEFRONT);
      e >>= 16;
      __hip_atomic_fetch_or((LAS unsigned*)(ll + 4096 + ((e >> 4) & 0x3fcu)), (e & 3u) << ((e >> 1) & 30u), __ATOMIC_RELAXED, __HIP_MEMORY_SCOPE_WAVEFRONT);
      return *(LAS volatile v4u*)(ll + 4096 + l16);
    };
    const unsigned base = call * 64;
    auto iter = [&](v4u (&R)[4], int j) {
      v4u pl = plane(j);
      const unsigned Bb = __builtin_amdgcn_readlane(hdr, j);
      if (j == NPUSH) asm volatile("s_waitcnt vmcnt(4)" : "+v"(R[0]), "+v"(R[1]), "+v"(R[2]), "+v"(R[3]), "+v"(pl) :: "memory");
      else if (j == NPUSH + 1) asm volatile("s_waitcnt vmcnt(9)" : "+v"(R[0]), "+v"(R[1]), "+v"(R[2]), "+v"(R[3]), "+v"(pl) :: "memory");
      else asm volatile("s_waitcnt vmcnt(10)" : "+v"(R[0]), "+v"(R[1]), "+v"(R[2]), "+v"(R[3]), "+v"(pl) :: "memory");
      v4u Y[4];
      #pragma unroll
      for (int e = 0; e < 4; ++e) {
        const unsigned q0 = R[0][e], q1 = R[1][e], q2 = R[2][e], p = pl[e] | R[3][e];
        const unsigned p3 = (__builtin_amdgcn_alignbit(q0, q0, 6) & (0x81u * REP)) | ((q1 >> 4) & (0x06u * REP)) | ((q2 >> 2) & (0x18u * REP));
        Y[0][e] = ((q0 & (0x9fu * REP)) + Bb) ^ ((p << 5) & (0x60u * REP));
        Y[1][e] = ((q1 & (0x9fu * REP)) + Bb) ^ ((p << 3) & (0x60u * REP));
        Y[2][e] = ((q2 & (0x9fu * REP)) + Bb) ^ ((p << 1) & (0x60u * REP));
        Y[3][e] = (p3 + Bb) ^ ((p >> 1) & (0x60u * REP));
      }
      asm volatile("global_store_dwordx4 %4, %0, %5 sc0 sc1\n\t"
                   "global_store_dwordx4 %4, %1, %5 offset:1024 sc0 sc1\n\t"
                   "global_store_dwordx4 %4, %2, %5 offset:2048 sc0 sc1\n\t"
                   "global_store_dwordx4 %4, %3, %5 offset:3072 sc0 sc1\n\ts_nop 1"
                   :: "v"(Y[0]), "v"(Y[1]), "v"(Y[2]), "v"(Y[3]), "v"(l16), "s"(rdst + (unsigned)j * PIECE) : "memory");
      LDM(R, j + 2);
      LDP(R, j + 2);
      asm volatile("s_waitcnt vmcnt(13)" ::: "memory");
      __builtin_amdgcn_s_barrier();
      if (lane < 8) {
        if (NPUSH > 0 && j == NPUSH) st_sys(wsp + floff + rgp * 4 * 32 * 128 + 4, base);
        else st_sys(flag, base + (unsigned)(j - NPUSH));
      }
      STAMP(8 + j);
    };
    iter(A, NPUSH);
    iter(B, NPUSH + 1);
    int j = NPUSH + 2;
    for (; j + 1 < NPIECE; j += 2) { iter(A, j); iter(B, j + 1); }
    if (j < NPIECE) iter(A, j);
    asm volatile("s_waitcnt vmcnt(1)" : "+v"(A[0]), "+v"(A[1]), "+v"(A[2]), "+v"(A[3]), "+v"(B[0]), "+v"(B[1]), "+v"(B[2]), "+v"(B[3]) :: "memory");
    __builtin_amdgcn_s_barrier();
    if (lane < 8) st_sys(flag, base + (unsigned)(NPIECE - NPUSH));
    STAMP(8 + NPIECE);
    STAMP(3);
    STAMP_OUT(0);
    return;
  }

  const int i = slot - 4;
  const int rg = i >> 2;
  const int hp = GMAP == 0 ? (i >> 1) & 1 : xcd & 1;
  const int cg = GMAP == 0 ? xcd * 2 + (i & 1) : (xcd >> 1) * 4 + (i & 3);
  const bool stager = rg < 4;
  if (stager) {
    const int task = (rg * 2 + hp) * 16 + cg;
    const int unit = task * 4 + wave;
    unsigned c; v4u X[4];
    const char* xs = P.x + (size_t)((task >> 2) * 16 + m) * K + (unit & 15) * 256 + g * 16;
    asm volatile("global_load_dwordx4 %0, %5, off\n\t"
                 "global_load_dwordx4 %1, %5, off offset:64\n\t"
                 "global_load_dwordx4 %2, %5, off offset:128\n\t"
                 "global_load_dwordx4 %3, %5, off offset:192\n\t"
                 "global_load_dword %4, %6, off sc0 sc1"
                 : "=&v"(X[0]), "=&v"(X[1]), "=&v"(X[2]), "=&v"(X[3]), "=&v"(c) : "v"(xs), "v"(cntp) : "memory");
    touch_w(slot);
    LAS char* ll = (LAS char*)smem + wave * 40960 + 4096;
    *(LAS volatile unsigned*)(ll + (lane & 31) * 4) = 0u;
    asm volatile("s_waitcnt vmcnt(%5)" : "+v"(c), "+v"(X[0]), "+v"(X[1]), "+v"(X[2]), "+v"(X[3]) : "i"(NT) : "memory");
    STAMP(6);
    call = __builtin_amdgcn_readfirstlane(c) + 1;
    unsigned sum = 0;
    #pragma unroll
    for (int e = 0; e < 4; ++e) sum = __builtin_amdgcn_sad_u8(X[0][e] & (0x7fu * REP), 0u, sum);
    sum += __builtin_amdgcn_update_dpp(0u, sum, 0x111, 0xf, 0xf, true);
    sum += __builtin_amdgcn_update_dpp(0u, sum, 0x112, 0xf, 0xf, true);
    sum += __builtin_amdgcn_update_dpp(0u, sum, 0x114, 0xf, 0xf, true);
    sum += __builtin_amdgcn_update_dpp(0u, sum, 0x118, 0xf, 0xf, true);
    int bs = (__builtin_amdgcn_readlane(sum, 15) + __builtin_amdgcn_readlane(sum, 31) + __builtin_amdgcn_readlane(sum, 47) + __builtin_amdgcn_readlane(sum, 63) - 20838) >> 10;
    bs = bs < 0 ? 0 : (bs > 96 ? 96 : bs);
    const unsigned Bb = (unsigned)bs * REP;
    v4u Q[3], PLq;
    #pragma unroll
    for (int e = 0; e < 4; ++e) {
      unsigned p[4], mk[4];
      #pragma unroll
      for (int i = 0; i < 4; ++i) {
        const unsigned x = X[i][e];
        p[i] = (((x | (0x80u * REP)) - Bb) & (0x1fu * REP)) | (x & (0x80u * REP));
        mk[i] = x ^ (p[i] + Bb);
      }
      Q[0][e] = p[0] | (__builtin_amdgcn_alignbit(p[3], p[3], 26) & (0x60u * REP));
      Q[1][e] = p[1] | ((p[3] << 4) & (0x60u * REP));
      Q[2][e] = p[2] | ((p[3] << 2) & (0x60u * REP));
      PLq[e] = (mk[0] >> 5) | (mk[1] >> 3) | (mk[2] >> 1) | (mk[3] << 1);
    }
    char* pk = wsl + OFF_PK + (call & 1) * PK_PAR;
    #pragma unroll
    for (int i = 0; i < 3; ++i) st16_sys(pk + unit * PKU + i * 1024 + lane * 16, Q[i]);
    #pragma unroll
    for (int i = 0; i < 4; ++i) st16_sys(wsl + OFF_RAW + unit * 4096 + i * 1024 + lane * 16, X[i]);
    int at = 0;
    #pragma unroll
    for (int e = 0; e < 4; ++e) at += __builtin_popcount((PLq[e] | (PLq[e] >> 1)) & 0x55555555u);
    int inc = at;
    inc += __builtin_amdgcn_update_dpp(0, inc, 0x111, 0xf, 0xf, true);
    inc += __builtin_amdgcn_update_dpp(0, inc, 0x112, 0xf, 0xf, true);
    inc += __builtin_amdgcn_update_dpp(0, inc, 0x114, 0xf, 0xf, true);
    inc += __builtin_amdgcn_update_dpp(0, inc, 0x118, 0xf, 0xf, true);
    inc += __builtin_amdgcn_update_dpp(0, inc, 0x142, 0xa, 0xf, false);
    inc += __builtin_amdgcn_update_dpp(0, inc, 0x143, 0xc, 0xf, false);
    const bool ovf = __builtin_amdgcn_readlane(inc, 63) > LCAP;
    if (!ovf) {
      LAS char* lp = ll + 4 + (inc - at) * 2;
      #pragma unroll
      for (int e = 0; e < 4; ++e) {
        unsigned t = PLq[e];
        while (t != 0) {
          const unsigned pos = (unsigned)__builtin_ctz(t) & 30u;
          *(LAS volatile unsigned short*)lp = (unsigned short)(((lane * 4 + e) << 6) | (pos << 1) | ((t >> pos) & 3u));
          lp += 2;
          t &= ~(3u << pos);
        }
      }
    }
    if (ovf) st16_sys(wsl + OFF_PL + (call & 1) * PL_PAR + unit * 1024 + lane * 16, PLq);
    unsigned le = *(LAS volatile unsigned*)(ll + (lane & 31) * 4);
    le = lane == 0 ? (unsigned)bs | (ovf ? 256u : 0u) : le;
    if (lane < 32) st_sys(pk + PK_LS + ((unit & 15) * 32 + (unit >> 4)) * 128 + lane * 4, le);
    STAMP(7);
    asm volatile("s_waitcnt vmcnt(0)" ::: "memory");
    __builtin_amdgcn_s_barrier();
    if (threadIdx.x == 0) st_sys(wsl + OFF_FLX + task * 128, call);
  } else {
    call = 0;
    touch_w(slot);
    if (slot >= 28) touch_w(slot - 28);
  }
  STAMP(1);

  const int rot = (rg + 2 * cg) & 7;
  LAS char* const wl = (LAS char*)smem + wave * 40960;
  v8i Wa[8][4], Wv[8][2];
  {
    v4u Hold[6][4];
    const int r8 = lane >> 3;
    const unsigned vb = (unsigned)(wave * 1024 + r8) * NFULL + rank * NS + cg * 128;
    const unsigned v0 = vb + (((lane & 7) ^ (r8 >> 1)) << 4), v1 = v0 ^ 64;
    auto issue = [&](int s) {
      #pragma unroll
      for (int q = 0; q < 8; ++q)
        asm volatile("s_mov_b32 m0, %0\n\ts_nop 0\n\tbuffer_load_dwordx4 %1, %2, %3 offen lds"
                     :: "s"(lds0 + wave * 40960 + 8192 + (s & 3) * 8192 + q * 1024), "v"((q & 2) ? v1 : v0), "s"(rw),
                        "s"((unsigned)(((((s >> 1) + rot) & 7) * 128 + (s & 1) * 64 + q * 8) * NFULL)) : "memory");
    };
    issue(0); issue(1); issue(2); issue(3);
    const int q8 = (lane & 15) >> 1, p = lane & 1;
    #pragma unroll
    for (int t = 0; t < 8; ++t) {
      v2i r[8][4];
      #pragma unroll
      for (int h = 0; h < 2; ++h) {
        const int s = 2 * t + h;
        if (s <= 12) asm volatile("s_waitcnt vmcnt(24)" ::: "memory");
        else if (s == 13) asm volatile("s_waitcnt vmcnt(16)" ::: "memory");
        else if (s == 14) asm volatile("s_waitcnt vmcnt(8)" ::: "memory");
        else asm volatile("s_waitcnt vmcnt(0)" ::: "memory");
        #pragma unroll
        for (int c = 0; c < 8; ++c)
          #pragma unroll
          for (int e = 0; e < 2; ++e) {
            const int row = 16 * g + 8 * e + q8;
            const int f = ((row >> 1) & 3) | (((row >> 4) & 1) << 2);
            r[c][2 * h + e] = __builtin_amdgcn_ds_read_tr8_b64_v2i32((LAS v2i*)(wl + 8192 + (s & 3) * 8192 + row * 128 + ((c ^ f) << 4) + p * 8));
          }
        __builtin_amdgcn_s_waitcnt(0xC07F);
        asm volatile("" ::: "memory");
        if (s < 12) issue(s + 4);
      }
      #pragma unroll
      for (int c = 0; c < 8; ++c) {
        if (c < 2) {
          const v4u lo = {(unsigned)r[c][0].x, (unsigned)r[c][0].y, (unsigned)r[c][1].x, (unsigned)r[c][1].y};
          const v4u hi = {(unsigned)r[c][2].x, (unsigned)r[c][2].y, (unsigned)r[c][3].x, (unsigned)r[c][3].y};
          if (t < 2) { *(LAS v4u*)(wl + t * 4096 + c * 2048 + lane * 16) = lo; *(LAS v4u*)(wl + t * 4096 + c * 2048 + 1024 + lane * 16) = hi; }
          else { Hold[t - 2][2 * c] = lo; Hold[t - 2][2 * c + 1] = hi; }
        } else {
          const v8i w = {r[c][0].x, r[c][0].y, r[c][1].x, r[c][1].y, r[c][2].x, r[c][2].y, r[c][3].x, r[c][3].y};
          if (c < 4) { Wv[t][c - 2] = w; asm volatile("" : "+v"(Wv[t][c - 2])); }
          else { Wa[t][c - 4] = w; asm volatile("" : "+a"(Wa[t][c - 4])); }
        }
      }
    }
    #pragma unroll
    for (int t = 2; t < 8; ++t)
      #pragma unroll
      for (int k = 0; k < 4; ++k) *(LAS v4u*)(wl + t * 4096 + k * 1024 + lane * 16) = Hold[t - 2][k];
  }
  SB;
  STAMP(2);
  if (!stager) call = __builtin_amdgcn_readfirstlane(ld_sys(cntp)) + 1;
  if (lane == 0) st_sys(cntp, call);

  float scale = __builtin_bit_cast(float, __builtin_amdgcn_readfirstlane(__builtin_bit_cast(int, sx * sw)));
#define OWNS 2
  constexpr int REST = NPIECE - 8 * OWNS;
  const int r2 = rg * 2 + hp;
  const int own0 = stager ? r2 * OWNS : 8 * OWNS + (REST * (r2 - 8)) / 6;
  const int nown = stager ? OWNS : 8 * OWNS + (REST * (r2 - 7)) / 6 - own0;
  const int np = nown + NPIECE / 2;
  const unsigned src = (rank + 1 + rg) & 7u;
  bool alive = true;
  {
    unsigned f;
    const int jj = own0 + ((lane & 7) < nown ? (lane & 7) : 0);
    if (!spin_ge(wsl + OFF_FLX + (jj * 4 + wave) * 128, call, f, wsl, call)) { alive = false; scale = __builtin_nanf(""); }
  }
  const int rj = (lane - nown) * 2 + hp;
  const unsigned t_x = lane < nown ? OFF_RAW + (unsigned)(own0 + lane) * PIECE
                     : rj < NPUSH ? OFF_PSH + (call & 1) * PSH_PAR + (unsigned)(rg * 4 + rj) * PIECE
                     : rj < NPIECE ? OFF_DST + rg * SHARD + (unsigned)rj * PIECE : OFF_END;
  const unsigned t_o = (lane < nown ? rank * MS + (unsigned)(own0 + lane) * 16 : src * MS + (unsigned)rj * 16) * (NS * 4);
  const unsigned t_f = (lane >= nown && rj < NPIECE) ? call * 64 + (unsigned)(rj < NPUSH ? 0 : rj + 1 - NPUSH) : 0u;
  const int npw = nown + (NPUSH + 1 - hp) / 2;
  __amdgpu_buffer_rsrc_t rs = __builtin_amdgcn_make_buffer_rsrc((void*)wsl, 0, (int)OFF_END, 0x00020000);
  __amdgpu_buffer_rsrc_t ro = __builtin_amdgcn_make_buffer_rsrc((void*)P.out, 0, M * NS * 4, 0x00020000);
  constexpr int AUX = 17;
  const unsigned voff = wave * 16384 + lane * 16;
  const char* const flagp = wsl + OFF_FLP + ((rg * 4 + wave) * 32 + hp * 16 + cg) * 128;
  __amdgpu_buffer_rsrc_t rf = __builtin_amdgcn_make_buffer_rsrc((void*)flagp, 0, 128, 0x00020000);
  v4u xb[8], wq[4];
  unsigned sbn = __builtin_amdgcn_readlane(t_x, 0);
  #pragma unroll
  for (int t = 0; t < 8; ++t) xb[t] = __builtin_amdgcn_raw_buffer_load_b128(rs, voff + (t & 1) * 1024, sbn + ((((t >> 1) + rot) & 7) << 11), AUX);
  #pragma unroll
  for (int k = 0; k < 4; ++k) wq[k] = *(LAS v4u*)(wl + k * 1024 + lane * 16);

  LAS char* rbw[4];
  #pragma unroll
  for (int k = 0; k < 4; ++k) rbw[k] = wl + 32768 + m * 512 + (((4 * k + g) ^ m) << 4);
  const int rr = 4 * wave + (lane >> 5);
  LAS char* rbr[2][2];
  #pragma unroll
  for (int hf = 0; hf < 2; ++hf) {
    rbr[hf][0] = (LAS char*)smem + 32768 + (rr + 2 * hf) * 512 + (((lane & 31) ^ (rr + 2 * hf)) << 4);
    rbr[hf][1] = rbr[hf][0] + 81920;
  }
  const unsigned ooff = (rr * NS + cg * 128 + (lane & 31) * 4) * 4;
  v4f acc[8];
#define PGAP 22
  auto step = [&](int n, bool prev, bool cur) __attribute__((always_inline)) {
    const unsigned sbc = sbn;
    unsigned fl = 0;
    v4f s0, s1, s2, s3;
    auto pw = [&](int c) { *(LAS v4f*)(rbw[c & 3] + (c >> 2) * 256) = acc[c]; };
    auto ldx = [&](int f, int half) {
      xb[2 * (f & 3) + half] = __builtin_amdgcn_raw_buffer_load_b128(rs, voff + half * 1024, (f < 8 ? sbc : sbn) + (((f + rot) & 7) << 11), AUX);
    };
    auto ldw = [&](int k, int t, int c) {
      wq[2 * k] = *(LAS v4u*)(wl + t * 4096 + c * 2048 + lane * 16);
      wq[2 * k + 1] = *(LAS v4u*)(wl + t * 4096 + c * 2048 + 1024 + lane * 16);
    };
    sfor<64>([&](auto G) __attribute__((always_inline)) {
      constexpr int gap = decltype(G)::value;
      constexpr bool pr = gap >= 48;
      constexpr int u = (gap - 48) >> 1;
      constexpr int t = pr ? 6 + (gap & 1) : gap >> 3, c = pr ? (u == 0 ? 0 : u < 4 ? u + 1 : u == 4 ? 1 : u) : gap & 7, k = pr ? gap & 1 : c & 1;
      if (prev && !cur && gap == 0) asm volatile("s_nop 7\n\ts_nop 7" ::: "memory");
      if (cur) {
        const v8i x = cat(xb[2 * (t & 3)], xb[2 * (t & 3) + 1]);
        if constexpr (c < 2) { const v8i w = cat(wq[2 * k], wq[2 * k + 1]); if (t == 0) mfma0v(acc[c], w, x); else mfmav(acc[c], w, x); }
        else if constexpr (c < 4) { if (t == 0) mfma0v(acc[c], Wv[t][c - 2], x); else mfmav(acc[c], Wv[t][c - 2], x); }
        else { if (t == 0) mfma0(acc[c], Wa[t][c - 4], x); else mfma(acc[c], Wa[t][c - 4], x); }
      }
      SB;
      if (cur) {
        if (gap < 40) { if (c == 2 || c == 3) ldw(c - 2, t + 1, c - 2); }
        else { if (gap == 42) ldw(0, 6, 0); if (gap == 43) ldw(1, 7, 0); if (gap == 50) ldw(0, 6, 1); if (gap == 52) ldw(1, 7, 1);
               if (gap == 58) ldw(0, 0, 0); if (gap == 60) ldw(1, 0, 1); }
        if (gap == PGAP) fl = __builtin_amdgcn_raw_buffer_load_b32(rf, 0, n + 1 < npw ? 4 : 0, AUX);
        if (gap == 38) {
          const unsigned target = __builtin_amdgcn_readlane(t_f, n + 1);
          sbn = __builtin_amdgcn_readlane(t_x, n + 1);
          if ((unsigned)__builtin_amdgcn_readfirstlane(fl) < target && alive) {
            unsigned f;
            if (!spin_ge(flagp + (n + 1 < npw ? 4 : 0), target, f, wsl, call)) { alive = false; scale = __builtin_nanf(""); }
            STAMP(8 + (n + 1 - nown) * 2 + hp + 1);
          }
        }
        if (gap < 48 && c == 7) ldx(t + 4, 0);
        if (gap >= 8 && gap <= 48 && (gap & 7) == 0) ldx((gap >> 3) + 3, 1);
        if (gap == 62) ldx(10, 0);
        if (prev) { if (gap == 0) ldx(2, 1); if (gap == 4) ldx(3, 0); if (gap == 5) ldx(3, 1); }
        if (gap == 51) pw(0); if (gap == 53) pw(2); if (gap == 55) pw(3); if (gap == 57) pw(4);
        if (gap == 59) pw(1); if (gap == 61) pw(5); if (gap == 63) pw(6);
      }
      if (prev) {
        if (gap == 1) pw(7);
        if (gap == 6) { asm volatile("s_waitcnt lgkmcnt(0)" ::: "memory"); __builtin_amdgcn_s_barrier(); asm volatile("" ::: "memory"); }
        #pragma unroll
        for (int hf = 0; hf < 2; ++hf) {
          if (gap == (hf ? 28 : 17)) { s0 = *(LAS v4f*)(rbr[hf][0]); s1 = *(LAS v4f*)(rbr[hf][0] + 40960); }
          if (gap == (hf ? 29 : 20)) { s2 = *(LAS v4f*)(rbr[hf][1]); s3 = *(LAS v4f*)(rbr[hf][1] + 40960); }
          if (gap == (hf ? 33 : 21)) { for (int e = 0; e < 4; ++e) asm("v_add_f32 %0, %1, %2" : "=v"(s0[e]) : "v"(s0[e]), "v"(s1[e])); }
          if (gap == (hf ? 34 : 24)) { for (int e = 0; e < 4; ++e) asm("v_add_f32 %0, %1, %2" : "=v"(s2[e]) : "v"(s2[e]), "v"(s3[e])); }
          if (gap == (hf ? 35 : 25)) { for (int e = 0; e < 4; ++e) asm("v_add_f32 %0, %1, %2" : "=v"(s0[e]) : "v"(s0[e]), "v"(s2[e])); }
          if (gap == (hf ? 36 : 26)) { for (int e = 0; e < 4; ++e) asm("v_mul_f32 %0, %1, %2" : "=v"(s0[e]) : "s"(scale), "v"(s0[e])); }
          if (gap == (hf ? 37 : 27)) __builtin_amdgcn_raw_buffer_store_b128(__builtin_bit_cast(v4u, s0), ro, ooff, __builtin_amdgcn_readlane(t_o, n - 1) + hf * 2 * NS * 4, 0);
        }
        if (gap == 46 && cur) { __builtin_amdgcn_s_barrier(); asm volatile("" ::: "memory"); }
      }
      SB;
    });
  };
  step(0, false, true);
  STAMP(4);
  for (int n = 1; n < np; ++n) {
    step(n, true, true);
    STAMP(43 + n);
  }
  step(np, true, false);
  STAMP(3);
  STAMP_OUT(0);
}

}

extern "C" void kernel_launch(int rank, int nranks, ncclComm_t comm, hipStream_t stream,
                              void* const* d_in, const int* in_sizes, int n_in,
                              void* d_out, int out_size,
                              const int* in_dtypes, int out_dtype,
                              void* const* d_out_all,
                              void* const* d_ws_all, size_t ws_bytes) {
  (void)comm; (void)in_dtypes; (void)out_dtype; (void)d_out_all;
  if (nranks != NR || n_in != 4 || (unsigned)rank >= (unsigned)NR || ws_bytes < WS_USED ||
      in_sizes[0] != MS * K || in_sizes[1] != K * NFULL || out_size != M * NS) {
    fprintf(stderr, "kernel_launch: unexpected problem shape\n");
    return;
  }
  HIP_CHECK(hipFuncSetAttribute((const void*)fused_ag_gemm, hipFuncAttributeMaxDynamicSharedMemorySize, LDS_BYTES));
  Params P;
  for (int r = 0; r < NR; ++r) P.ws[r] = (char*)d_ws_all[(rank + 1 + r) % NR];
  P.wsl = (char*)d_ws_all[rank];
  P.x = (const char*)d_in[0];
  P.w = (const char*)d_in[1];
  P.sx = (const float*)d_in[2];
  P.sw = (const float*)d_in[3];
  P.out = (float*)d_out;
  P.rank = rank;
  P.pad = 0;
  fused_ag_gemm<<<NWG, 256, LDS_BYTES, stream>>>(P);
  HIP_CHECK(hipGetLastError());
}
```

```cpp
#include <hip/hip_runtime.h>
#include <rccl/rccl.h>
#include <cstdio>
#include <cstdint>
#include <type_traits>
#include <utility>

#define HIP_CHECK(expr) do { hipError_t _e = (expr); if (_e != hipSuccess) { \
    fprintf(stderr, "HIP error %s at %s:%d\n", hipGetErrorString(_e), __FILE__, __LINE__); \
    return; } } while (0)

typedef unsigned v4u __attribute__((ext_vector_type(4)));
typedef int v8i __attribute__((ext_vector_type(8)));
typedef int v2i __attribute__((ext_vector_type(2)));
typedef float v4f __attribute__((ext_vector_type(4)));
#define LAS __attribute__((address_space(3)))
#define SB __builtin_amdgcn_sched_barrier(0)

namespace {

constexpr int NR = 8;
constexpr int M = 4096, K = 4096, NFULL = 16384, NS = 2048, MS = 512;
constexpr unsigned SHARD = MS * K;
constexpr unsigned PIECE = 16 * K;
constexpr int NPIECE = MS / 16;
constexpr int NRG = 7;
constexpr int NWG = 256;
constexpr int FWD_XCD = 3;
constexpr int LDS_BYTES = 163840;

#define NPUSH 3
#define KDRAIN 6
#define SWAPH 0
#define GROUP_MASK 0x4D
constexpr unsigned PKU = 3072;
constexpr int LCAP = 62;

constexpr unsigned OFF_RAW = 0;
constexpr unsigned OFF_DST = SHARD;
constexpr unsigned OFF_PSH = 8 * SHARD;
constexpr unsigned PSH_PAR = SHARD;
constexpr unsigned OFF_END = 10 * SHARD;
constexpr unsigned OFF_PK  = OFF_END;
constexpr unsigned PK_PAR = SHARD, PK_LS = 512 * PKU, PK_PAD = PK_LS + 512 * 128;
constexpr unsigned OFF_PL  = 12 * SHARD;
constexpr unsigned PL_PAR = 512 * 1024;
constexpr unsigned OFF_FLX = 13 * SHARD;
constexpr unsigned OFF_FGO = OFF_FLX + 128 * 128;
constexpr unsigned OFF_FLP = OFF_FGO + NRG * 4 * 128;
constexpr unsigned OFF_CNT = OFF_FLP + NRG * 4 * 32 * 128;
constexpr unsigned OFF_TMO = OFF_CNT + NWG * 128;
constexpr unsigned OFF_PAD = OFF_TMO + 128;
constexpr unsigned OFF_DBG = OFF_PAD + 64 * 128;
constexpr unsigned OFF_DB2 = OFF_DBG + NWG * 64 * 4;
constexpr unsigned WS_USED = OFF_DB2 + 4096 * 64 * 4;
constexpr unsigned long long DEADLINE = 30000000ull;
static_assert(NPUSH >= 0 && NPUSH <= 3 && 7 * 4 * PIECE <= PSH_PAR && PK_PAD + 64 * 128 <= PK_PAR, "layout");

struct Params {
  char* ws[NR];
  char* wsl;
  const char* x;
  const char* w;
  const float* sx;
  const float* sw;
  float* out;
  int rank;
  unsigned peers;
  unsigned back;
  int pad;
};

#define STAMP(i) do {} while (0)
#define STAMP_OUT(w) do {} while (0)

__device__ __forceinline__ void mfma0(v4f& acc, const v8i& w, const v8i& x) {
  asm volatile("v_mfma_f32_16x16x128_f8f6f4 %0, %1, %2, 0 cbsz:1 blgp:1" : "=&v"(acc) : "a"(w), "v"(x));
}
__device__ __forceinline__ void mfma(v4f& acc, const v8i& w, const v8i& x) {
  asm volatile("v_mfma_f32_16x16x128_f8f6f4 %0, %1, %2, %0 cbsz:1 blgp:1" : "+v"(acc) : "a"(w), "v"(x));
}
__device__ __forceinline__ void mfma0v(v4f& acc, const v8i& w, const v8i& x) {
  asm volatile("v_mfma_f32_16x16x128_f8f6f4 %0, %1, %2, 0 cbsz:1 blgp:1" : "=&v"(acc) : "v"(w), "v"(x));
}
__device__ __forceinline__ void mfmav(v4f& acc, const v8i& w, const v8i& x) {
  asm volatile("v_mfma_f32_16x16x128_f8f6f4 %0, %1, %2, %0 cbsz:1 blgp:1" : "+v"(acc) : "v"(w), "v"(x));
}
__device__ __forceinline__ v8i cat(const v4u& lo, const v4u& hi) {
  return v8i{(int)lo.x, (int)lo.y, (int)lo.z, (int)lo.w, (int)hi.x, (int)hi.y, (int)hi.z, (int)hi.w};
}
template <class F, int... Is> __device__ __forceinline__ void sfor_impl(F&& f, std::integer_sequence<int, Is...>) { (f(std::integral_constant<int, Is>{}), ...); }
template <int N, class F> __device__ __forceinline__ void sfor(F&& f) { sfor_impl(f, std::make_integer_sequence<int, N>{}); }
__device__ __forceinline__ unsigned ld_sys(const void* p) {
  unsigned v;
  asm volatile("global_load_dword %0, %1, off sc0 sc1\n\ts_waitcnt vmcnt(0)" : "=&v"(v) : "v"(p) : "memory");
  return v;
}
__device__ __forceinline__ void st_sys(void* p, unsigned v) {
  asm volatile("global_store_dword %0, %1, off sc0 sc1" :: "v"(p), "v"(v) : "memory");
}
__device__ __forceinline__ bool spin_ge(const void* p, unsigned target, unsigned& f, char* wsl, unsigned call) {
  unsigned long long t0 = __builtin_amdgcn_s_memrealtime();
  for (unsigned n = 0;; ++n) {
    f = ld_sys(p);
    if (__builtin_amdgcn_ballot_w64(f < target) == 0) return true;
    if ((n & 63) == 63) {
      unsigned t = ld_sys(wsl + OFF_TMO);
      if (__builtin_amdgcn_readfirstlane(t) == call || __builtin_amdgcn_s_memrealtime() - t0 > DEADLINE) {
        st_sys(wsl + OFF_TMO, call);
        return false;
      }
    }
    __builtin_amdgcn_s_sleep(1);
  }
}

__device__ __forceinline__ void st16_sys(void* p, const v4u& v) {
  asm volatile("global_store_dwordx4 %0, %1, off sc0 sc1\n\ts_nop 1" :: "v"(p), "v"(v) : "memory");
}

constexpr unsigned REP = 0x01010101u;
__device__ __forceinline__ unsigned bfi(unsigned mask, unsigned a, unsigned b) { return __builtin_amdgcn_bitop3_b32(mask, a, b, 0xCA); }
__device__ __forceinline__ unsigned rotr(unsigned v, unsigned n) { return __builtin_amdgcn_alignbit(v, v, n); }

__global__ void __launch_bounds__(256, 1) fused_ag_gemm(Params P) {
  extern __shared__ __attribute__((aligned(16))) char smem[];
  const int b = blockIdx.x, xcd = b & 7, slot = b >> 3;
  const int wave = __builtin_amdgcn_readfirstlane(threadIdx.x >> 6), lane = threadIdx.x & 63;
  const int g = lane >> 4, m = lane & 15;
  const unsigned rank = (unsigned)P.rank & 7u;
  char* const wsl = P.wsl;
  const int cpi = xcd < FWD_XCD ? xcd : xcd - 1;
  char* const wsp = P.ws[cpi];
  float sx = P.sx[0], sw = P.sw[0];
  STAMP(0);
  asm volatile("" :: "s"(P.x), "s"(P.w), "s"(P.out), "s"(wsl), "s"(wsp), "s"(rank), "s"(P.peers), "s"(P.back));

  char* const cntp = wsl + OFF_CNT + b * 128 + wave * 4;
  const unsigned lds0 = (unsigned)(size_t)(LAS char*)smem;
  __amdgpu_buffer_rsrc_t rw = __builtin_amdgcn_make_buffer_rsrc((void*)P.w, 0, K * NFULL, 0x00020000);
  auto touch_w = [&](unsigned id, int half) {
    const unsigned row = (id >> 11) * 1024 + (half ^ (xcd & SWAPH)) * 512 + ((id >> 2) & 511);
    const unsigned off = row * NFULL + rank * NS + (xcd >> 1) * 512 + (id & 3) * 128;
    asm volatile("s_mov_b32 m0, %0\n\ts_nop 0\n\tbuffer_load_dword %1, %2, 0 offen lds"
                 :: "s"(lds0 + wave * 40960), "v"(off), "s"(rw) : "memory");
  };
  auto touch = [&](const void* p) {
    asm volatile("s_mov_b32 m0, %0\n\ts_nop 0\n\tglobal_load_lds_dword %1, off sc0 sc1"
                 :: "s"(lds0 + wave * 40960), "v"(p) : "memory");
  };
  unsigned call;

  if (slot < 4) {
    const int q = slot;
    __builtin_amdgcn_s_waitcnt(0x0F70);
    if (xcd == FWD_XCD) {
      if (wave != 0) return;
      char* peer = P.ws[0];
      #pragma unroll
      for (int r = 1; r < NRG; ++r) peer = lane == r ? P.ws[r] : peer;
      const int lb = (P.back >> (4 * (lane < NRG ? lane : 0))) & 7;
      call = __builtin_amdgcn_readfirstlane(ld_sys(cntp)) + 1;
      touch(peer + OFF_PAD + (32 + lb * 4 + q) * 128);
      if (lane == 0) st_sys(cntp, call);
      unsigned f;
      if (!spin_ge(wsl + OFF_FLX + ((lane & 31) * 4 + q) * 128, call, f, wsl, call)) return;
      STAMP(2);
      if (lane < NRG) st_sys(peer + OFF_FGO + (lb * 4 + q) * 128, call);
      STAMP_OUT(0);
      return;
    }
    const int rg = cpi, rgp = (P.back >> (4 * cpi)) & 7;
    bool ok = true;
    if (wave < NPUSH) {
      v4u X[16]; unsigned c;
      const char* xs = P.x + (size_t)(wave * 16 + m) * K + q * 1024 + g * 16;
      asm volatile("global_load_dword %0, %1, off sc0 sc1" : "=&v"(c) : "v"(cntp) : "memory");
      #pragma unroll
      for (int i = 0; i < 16; ++i) asm volatile("global_load_dwordx4 %0, %1, off offset:%2" : "=&v"(X[i]) : "v"(xs), "i"(i * 64) : "memory");
      asm volatile("s_waitcnt vmcnt(15)" : "+v"(c), "+v"(X[0]) :: "memory");
      call = __builtin_amdgcn_readfirstlane(c) + 1;
      STAMP(6);
      char* d = wsp + OFF_PSH + (call & 1) * PSH_PAR + (rgp * 4 + wave) * PIECE + q * 16384;
      #pragma unroll
      for (int i = 0; i < 16; ++i)
        asm volatile("s_waitcnt vmcnt(15)\n\tglobal_store_dwordx4 %0, %1, %2 offset:%3 sc0 sc1\n\ts_nop 1"
                     :: "v"(lane * 16), "v"(X[i]), "s"(d + (i >> 2) * 4096), "i"((i & 3) * 1024) : "memory");
      STAMP(1);
      asm volatile("s_waitcnt vmcnt(%0)" :: "i"(KDRAIN) : "memory");
      STAMP(7);
      if (lane == 0) st_sys(cntp, call);
    } else {
      if (lane < 2) touch(wsp + OFF_PK + lane * PK_PAR + PK_PAD + (rgp * 4 + q) * 128);
      call = __builtin_amdgcn_readfirstlane(ld_sys(cntp)) + 1;
      if (lane == 0) st_sys(cntp, call);
      if (wave == 3) {
        unsigned f;
        ok = spin_ge(wsl + OFF_FGO + (rg * 4 + q) * 128, call, f, wsl, call);
        *(LAS volatile unsigned*)(smem + 4096) = ok ? 1u : 0u;
        asm volatile("s_waitcnt lgkmcnt(0)" ::: "memory");
      }
    }
    __builtin_amdgcn_s_barrier();
    asm volatile("" ::: "memory");
    STAMP(2);

    const int v = q * 4 + wave;
    const char* pk = wsp + OFF_PK + (call & 1) * PK_PAR;
    __amdgpu_buffer_rsrc_t rpk = __builtin_amdgcn_make_buffer_rsrc((void*)pk, 0, (int)PK_LS, 0x00020000);
    __amdgpu_buffer_rsrc_t rpl = __builtin_amdgcn_make_buffer_rsrc((void*)(wsp + OFF_PL + (call & 1) * PL_PAR), 0, (int)PL_PAR, 0x00020000);
    char* rdst = wsl + OFF_DST + rg * SHARD + v * 4096;
    const unsigned floff = OFF_FLP + (q * 32 + wave * 8 + (lane & 7)) * 128;
    char* flag = wsl + floff + rg * 4 * 32 * 128;
    LAS char* ll = (LAS char*)smem + wave * 40960 + 8192;
    const unsigned l16 = lane * 16;
    if (__builtin_amdgcn_readfirstlane(*(LAS volatile unsigned*)(smem + 4096)) == 0) return;
    v4u A[4], B[4];
    #define LDM(R, j) asm volatile("buffer_load_dwordx4 %0, %3, %4, %5 offen sc0 sc1\n\t" \
                                   "buffer_load_dwordx4 %1, %3, %4, %5 offen offset:1024 sc0 sc1\n\t" \
                                   "buffer_load_dwordx4 %2, %3, %4, %5 offen offset:2048 sc0 sc1" \
                                   : "=&v"(R[0]), "=&v"(R[1]), "=&v"(R[2]) : "v"(l16), "s"(rpk), "s"((unsigned)((j) * 16 + v) * PKU) : "memory")
    #define LDP(R, j) asm volatile("s_nop 4\n\tbuffer_load_dwordx4 %0, %1, %2, %3 offen sc0 sc1" : "=&v"(R[3]) : "v"(l16), "s"(rpl), \
                                   "s"(__builtin_amdgcn_readlane(plo, (j))) : "memory")
    #pragma unroll
    for (int i = 0; i < 4; ++i)
      asm volatile("s_mov_b32 m0, %0\n\ts_nop 0\n\tglobal_load_lds_dwordx4 %1, off sc0 sc1"
                   :: "s"(lds0 + wave * 40960 + 8192 + i * 1024), "v"(pk + PK_LS + v * 4096 + i * 1024 + l16) : "memory");
    LDM(A, NPUSH);
    asm volatile("s_waitcnt vmcnt(3)" ::: "memory");
    unsigned hdr = *(LAS volatile unsigned*)(ll + (lane & 31) * 128);
    const unsigned plo = (lane < 32 && (hdr & 256u)) ? (unsigned)(lane * 16 + v) * 1024u : PL_PAR;
    hdr = (hdr & 127u) * REP;
    LDP(A, NPUSH);
    LDM(B, NPUSH + 1);
    LDP(B, NPUSH + 1);
    auto plane1 = [&](int j) -> unsigned {
      *(LAS volatile v4u*)(ll + 4096 + l16) = v4u{0u, 0u, 0u, 0u};
      return *(LAS volatile unsigned short*)(ll + (j & 31) * 128 + lane * 2);
    };
    auto plane2 = [&](unsigned e) -> v4u {
      if (lane >= 2 && (e & 3u))
        __hip_atomic_fetch_or((LAS unsigned*)(ll + 4096 + ((e >> 4) & 0x3fcu)), rotr((e & 3u) << ((e >> 1) & 30u), 31), __ATOMIC_RELAXED, __HIP_MEMORY_SCOPE_WAVEFRONT);
      return *(LAS volatile v4u*)(ll + 4096 + l16);
    };
    const unsigned base = call * 64;
    v4u pl = plane2(plane1(NPUSH));
    auto iter = [&](v4u (&R)[4], int j) {
      const unsigned Bb = __builtin_amdgcn_readlane(hdr, j);
      if (j == NPUSH) asm volatile("s_waitcnt vmcnt(4)" : "+v"(R[0]), "+v"(R[1]), "+v"(R[2]), "+v"(R[3]), "+v"(pl) :: "memory");
      else if (j == NPUSH + 1) asm volatile("s_waitcnt vmcnt(9)" : "+v"(R[0]), "+v"(R[1]), "+v"(R[2]), "+v"(R[3]), "+v"(pl) :: "memory");
      else asm volatile("s_waitcnt vmcnt(10)" : "+v"(R[0]), "+v"(R[1]), "+v"(R[2]), "+v"(R[3]), "+v"(pl) :: "memory");
      const unsigned en = plane1(j + 1);
      char* const d = rdst + (unsigned)j * PIECE;
      v4u p = pl | R[3], Y;
      #define ST1(o) asm volatile("global_store_dwordx4 %0, %1, %2 offset:" #o " sc0 sc1\n\ts_nop 1" :: "v"(l16), "v"(Y), "s"(d) : "memory")
      #pragma unroll
      for (int e = 0; e < 4; ++e) Y[e] = bfi(0x60u * REP, rotr(p[e], 26), R[0][e]) + Bb;
      ST1(0);
      #pragma unroll
      for (int e = 0; e < 4; ++e) Y[e] = bfi(0x60u * REP, p[e] << 4, R[1][e]) + Bb;
      ST1(1024);
      #pragma unroll
      for (int e = 0; e < 4; ++e) Y[e] = bfi(0x60u * REP, p[e] << 2, R[2][e]) + Bb;
      ST1(2048);
      #pragma unroll
      for (int e = 0; e < 4; ++e) Y[e] = bfi(0x81u * REP, rotr(R[0][e], 6), bfi(0x06u * REP, R[1][e] >> 4, bfi(0x18u * REP, R[2][e] >> 2, p[e]))) + Bb;
      ST1(3072);
      LDM(R, j + 2);
      LDP(R, j + 2);
      pl = plane2(en);
      asm volatile("s_waitcnt vmcnt(13)" ::: "memory");
      __builtin_amdgcn_s_barrier();
      if (lane < 8) {
        if (NPUSH > 0 && j == NPUSH) st_sys(wsp + floff + rgp * 4 * 32 * 128 + 4, base);
        else st_sys(flag, base + (unsigned)(j - NPUSH));
      }
      STAMP(8 + j);
    };
    iter(A, NPUSH);
    iter(B, NPUSH + 1);
    int j = NPUSH + 2;
    for (; j + 1 < NPIECE; j += 2) { iter(A, j); iter(B, j + 1); }
    if (j < NPIECE) iter(A, j);
    asm volatile("s_waitcnt vmcnt(1)" : "+v"(A[0]), "+v"(A[1]), "+v"(A[2]), "+v"(A[3]), "+v"(B[0]), "+v"(B[1]), "+v"(B[2]), "+v"(B[3]) :: "memory");
    __builtin_amdgcn_s_barrier();
    if (lane < 8) st_sys(flag, base + (unsigned)(NPIECE - NPUSH));
    STAMP(8 + NPIECE);
    STAMP(3);
    STAMP_OUT(0);
    return;
  }

  const int i = slot - 4;
  const int rg = i >> 2;
  const int hp = xcd & 1;
  const int cg = (xcd >> 1) * 4 + (i & 3);
  const bool stager = rg < 4;
  if (stager) {
    const int task = (rg * 2 + hp) * 16 + cg;
    const int unit = task * 4 + wave;
    unsigned c; v4u X[4];
    const char* xs = P.x + (size_t)((task >> 2) * 16 + m) * K + (unit & 15) * 256 + g * 16;
    asm volatile("global_load_dwordx4 %0, %5, off\n\t"
                 "global_load_dwordx4 %1, %5, off offset:64\n\t"
                 "global_load_dwordx4 %2, %5, off offset:128\n\t"
                 "global_load_dwordx4 %3, %5, off offset:192\n\t"
                 "global_load_dword %4, %6, off sc0 sc1"
                 : "=&v"(X[0]), "=&v"(X[1]), "=&v"(X[2]), "=&v"(X[3]), "=&v"(c) : "v"(xs), "v"(cntp) : "memory");
#define EARLYT 0
    if (EARLYT == 1) touch_w(lane < 32 ? 6144 + i * 128 + wave * 32 + lane : 8192, 0);
    if (EARLYT == 2) touch_w(4096 + i * 256 + threadIdx.x, 0);
    LAS char* ll = (LAS char*)smem + wave * 40960 + 4096;
    *(LAS volatile unsigned*)(ll + (lane & 31) * 4) = 0u;
    asm volatile("s_waitcnt vmcnt(%5)" : "+v"(c), "+v"(X[0]), "+v"(X[1]), "+v"(X[2]), "+v"(X[3]) : "i"(EARLYT ? 1 : 0) : "memory");
    STAMP(6);
    call = __builtin_amdgcn_readfirstlane(c) + 1;
    unsigned sum = 0;
    #pragma unroll
    for (int e = 0; e < 4; ++e) sum = __builtin_amdgcn_sad_u8(X[0][e] & (0x7fu * REP), 0u, sum);
    sum += __builtin_amdgcn_update_dpp(0u, sum, 0x111, 0xf, 0xf, true);
    sum += __builtin_amdgcn_update_dpp(0u, sum, 0x112, 0xf, 0xf, true);
    sum += __builtin_amdgcn_update_dpp(0u, sum, 0x114, 0xf, 0xf, true);
    sum += __builtin_amdgcn_update_dpp(0u, sum, 0x118, 0xf, 0xf, true);
    int bs = (__builtin_amdgcn_readlane(sum, 15) + __builtin_amdgcn_readlane(sum, 31) + __builtin_amdgcn_readlane(sum, 47) + __builtin_amdgcn_readlane(sum, 63) - 20838) >> 10;
    bs = bs < 0 ? 0 : (bs > 96 ? 96 : bs);
    const unsigned Bb = (unsigned)bs * REP;
    v4u Q[3], PLq;
    #pragma unroll
    for (int e = 0; e < 4; ++e) {
      const unsigned t0 = X[0][e] - Bb, t1 = X[1][e] - Bb, t2 = X[2][e] - Bb, t3 = X[3][e] - Bb;
      Q[0][e] = bfi(0x60u * REP, rotr(t3, 26), t0);
      Q[1][e] = bfi(0x60u * REP, t3 << 4, t1);
      Q[2][e] = bfi(0x60u * REP, t3 << 2, t2);
      PLq[e] = bfi(0x81u * REP, rotr(t0, 6), bfi(0x06u * REP, t1 >> 4, bfi(0x18u * REP, t2 >> 2, t3)));
    }
    char* pk = wsl + OFF_PK + (call & 1) * PK_PAR;
    #pragma unroll
    for (int i = 0; i < 3; ++i) st16_sys(pk + unit * PKU + i * 1024 + lane * 16, Q[i]);
    #pragma unroll
    for (int i = 0; i < 4; ++i) st16_sys(wsl + OFF_RAW + unit * 4096 + i * 1024 + lane * 16, X[i]);
    int at = 0;
    #pragma unroll
    for (int e = 0; e < 4; ++e) at += __builtin_popcount((PLq[e] | rotr(PLq[e], 1)) & 0xaaaaaaaau);
    int inc = at;
    inc += __builtin_amdgcn_update_dpp(0, inc, 0x111, 0xf, 0xf, true);
    inc += __builtin_amdgcn_update_dpp(0, inc, 0x112, 0xf, 0xf, true);
    inc += __builtin_amdgcn_update_dpp(0, inc, 0x114, 0xf, 0xf, true);
    inc += __builtin_amdgcn_update_dpp(0, inc, 0x118, 0xf, 0xf, true);
    inc += __builtin_amdgcn_update_dpp(0, inc, 0x142, 0xa, 0xf, false);
    inc += __builtin_amdgcn_update_dpp(0, inc, 0x143, 0xc, 0xf, false);
    const bool ovf = __builtin_amdgcn_readlane(inc, 63) > LCAP;
    if (!ovf) {
      LAS char* lp = ll + 4 + (inc - at) * 2;
      #pragma unroll
      for (int e = 0; e < 4; ++e) {
        unsigned t = rotr(PLq[e], 1);
        while (t != 0) {
          const unsigned pos = (unsigned)__builtin_ctz(t) & 30u;
          *(LAS volatile unsigned short*)lp = (unsigned short)(((lane * 4 + e) << 6) | (pos << 1) | ((t >> pos) & 3u));
          lp += 2;
          t &= ~(3u << pos);
        }
      }
    }
    if (ovf) st16_sys(wsl + OFF_PL + (call & 1) * PL_PAR + unit * 1024 + lane * 16, PLq);
    unsigned le = *(LAS volatile unsigned*)(ll + (lane & 31) * 4);
    le = lane == 0 ? (unsigned)bs | (ovf ? 256u : 0u) : le;
    if (lane < 32) st_sys(pk + PK_LS + ((unit & 15) * 32 + (unit >> 4)) * 128 + lane * 4, le);
    STAMP(7);
    #pragma unroll
    for (int k = 0; k < 2; ++k) touch_w(k * 4096 + i * 256 + threadIdx.x, 1);
  } else {
    call = 0;
    if (EARLYT == 2) {
      touch_w((i - 16) * 256 + threadIdx.x, 0);
      touch_w(lane < 22 && (i - 16) * 88 + wave * 22 + lane < 1024 ? 3072 + (i - 16) * 88 + wave * 22 + lane : 8192, 0);
    } else {
      #pragma unroll
      for (int k = 0; k < 3 - EARLYT; ++k) touch_w(k * 3072 + (i - 16) * 256 + threadIdx.x, 0);
    }
  }

  const int rot = (cg + rg) & 3;
  auto kb = [&](int t) -> unsigned { return (unsigned)(((t & 4) ^ ((xcd & SWAPH) << 2)) | ((t + rot) & 3)); };
  LAS char* const wl = (LAS char*)smem + wave * 40960;
  v8i Wa[8][4], Wv[8][2];
  {
    v4u Hold[6][4];
    const int r8 = lane >> 3;
    const unsigned vb = (unsigned)(wave * 1024 + r8) * NFULL + rank * NS + cg * 128;
    const unsigned v0 = vb + (((lane & 7) ^ (r8 >> 1)) << 4), v1 = v0 ^ 64;
    auto issue = [&](int s, int q0 = 0, int q1 = 8) {
      #pragma unroll
      for (int q = q0; q < q1; ++q)
        asm volatile("s_mov_b32 m0, %0\n\ts_nop 0\n\tbuffer_load_dwordx4 %1, %2, %3 offen lds"
                     :: "s"(lds0 + wave * 40960 + 8192 + (s & 3) * 8192 + q * 1024), "v"((q & 2) ? v1 : v0), "s"(rw),
                        "s"((kb(s >> 1) * 128 + (s & 1) * 64 + q * 8) * NFULL) : "memory");
    };
#define NPRE 0
    issue(0, 0, NPRE);
    if (stager) {
      asm volatile("s_waitcnt vmcnt(%0)" :: "i"(NPRE + 2) : "memory");
      __builtin_amdgcn_s_barrier();
      if (threadIdx.x == 0) st_sys(wsl + OFF_FLX + ((rg * 2 + hp) * 16 + cg) * 128, call);
    }
    STAMP(1);
    issue(0, NPRE, 8); issue(1); issue(2);
    const int q8 = (lane & 15) >> 1, p = lane & 1;
    #pragma unroll
    for (int t = 0; t < 8; ++t) {
      v2i r[8][4];
      #pragma unroll
      for (int h = 0; h < 2; ++h) {
        const int s = 2 * t + h;
        if (s <= 13) asm volatile("s_waitcnt vmcnt(16)" ::: "memory");
        else if (s == 14) asm volatile("s_waitcnt vmcnt(8)" ::: "memory");
        else asm volatile("s_waitcnt vmcnt(0)" ::: "memory");
        if (s == 0) { STAMP(41); }
        #pragma unroll
        for (int c = 0; c < 8; ++c) {
          #pragma unroll
          for (int e = 0; e < 2; ++e) {
            const int row = 16 * g + 8 * e + q8;
            const int f = ((row >> 1) & 3) | (((row >> 4) & 1) << 2);
            r[c][2 * h + e] = __builtin_amdgcn_ds_read_tr8_b64_v2i32((LAS v2i*)(wl + 8192 + (s & 3) * 8192 + row * 128 + ((c ^ f) << 4) + p * 8));
          }
          if (s + 3 < 16) issue(s + 3, c, c + 1);
        }
        __builtin_amdgcn_s_waitcnt(0xC07F);
        asm volatile("" ::: "memory");
      }
      #pragma unroll
      for (int c = 0; c < 8; ++c) {
        if (c < 2) {
          const v4u lo = {(unsigned)r[c][0].x, (unsigned)r[c][0].y, (unsigned)r[c][1].x, (unsigned)r[c][1].y};
          const v4u hi = {(unsigned)r[c][2].x, (unsigned)r[c][2].y, (unsigned)r[c][3].x, (unsigned)r[c][3].y};
          if (t < 2) { *(LAS v4u*)(wl + t * 4096 + c * 2048 + lane * 16) = lo; *(LAS v4u*)(wl + t * 4096 + c * 2048 + 1024 + lane * 16) = hi; }
          else { Hold[t - 2][2 * c] = lo; Hold[t - 2][2 * c + 1] = hi; }
        } else {
          const v8i w = {r[c][0].x, r[c][0].y, r[c][1].x, r[c][1].y, r[c][2].x, r[c][2].y, r[c][3].x, r[c][3].y};
          if (c < 4) { Wv[t][c - 2] = w; asm volatile("" : "+v"(Wv[t][c - 2])); }
          else { Wa[t][c - 4] = w; asm volatile("" : "+a"(Wa[t][c - 4])); }
        }
      }
    }
    #pragma unroll
    for (int t = 2; t < 8; ++t)
      #pragma unroll
      for (int k = 0; k < 4; ++k) *(LAS v4u*)(wl + t * 4096 + k * 1024 + lane * 16) = Hold[t - 2][k];
  }
  SB;
  STAMP(2);
  if (!stager) call = __builtin_amdgcn_readfirstlane(ld_sys(cntp)) + 1;
  if (lane == 0) st_sys(cntp, call);

  float scale = __builtin_bit_cast(float, __builtin_amdgcn_readfirstlane(__builtin_bit_cast(int, sx * sw)));
#define OWNN 18
  const int r2 = rg * 2 + hp;
  const int own0 = stager ? ((NPIECE - OWNN) * r2) / 8 : NPIECE - OWNN + (OWNN * (r2 - 8)) / 6;
  const int nown = (stager ? ((NPIECE - OWNN) * (r2 + 1)) / 8 : NPIECE - OWNN + (OWNN * (r2 - 7)) / 6) - own0;
  static_assert(OWNN >= 6 && OWNN <= 24, "every workgroup needs at least one own piece");
  const int np = nown + NPIECE / 2;
  const unsigned src = (P.peers >> (4 * rg)) & 7u;
  bool alive = true;
  {
    unsigned f;
    const int jj = own0 + ((lane & 7) < nown ? (lane & 7) : 0);
    if (!spin_ge(wsl + OFF_FLX + (jj * 4 + wave) * 128, call, f, wsl, call)) { alive = false; scale = __builtin_nanf(""); }
  }
  const int rj = (lane - nown) * 2 + hp;
  const unsigned t_x = lane < nown ? OFF_RAW + (unsigned)(own0 + lane) * PIECE
                     : rj < NPUSH ? OFF_PSH + (call & 1) * PSH_PAR + (unsigned)(rg * 4 + rj) * PIECE
                     : rj < NPIECE ? OFF_DST + rg * SHARD + (unsigned)rj * PIECE : OFF_END;
  const unsigned t_o = (lane < nown ? rank * MS + (unsigned)(own0 + lane) * 16 : src * MS + (unsigned)rj * 16) * (NS * 4);
  const unsigned t_f = (lane >= nown && rj < NPIECE) ? call * 64 + (unsigned)(rj < NPUSH ? 0 : rj + 1 - NPUSH) : 0u;
  const int npw = nown + (NPUSH + 1 - hp) / 2;
  __amdgpu_buffer_rsrc_t rs = __builtin_amdgcn_make_buffer_rsrc((void*)wsl, 0, (int)OFF_END, 0x00020000);
  __amdgpu_buffer_rsrc_t ro = __builtin_amdgcn_make_buffer_rsrc((void*)P.out, 0, M * NS * 4, 0x00020000);
  constexpr int AUX = 17;
  const unsigned voff = wave * 16384 + lane * 16;
  const char* const flagp = wsl + OFF_FLP + ((rg * 4 + wave) * 32 + hp * 16 + cg) * 128;
  __amdgpu_buffer_rsrc_t rf = __builtin_amdgcn_make_buffer_rsrc((void*)flagp, 0, 128, 0x00020000);
  v4u xb[8], wq[4];
  unsigned sbn = __builtin_amdgcn_readlane(t_x, 0);
  #pragma unroll
  for (int t = 0; t < 8; ++t) xb[t] = __builtin_amdgcn_raw_buffer_load_b128(rs, voff + (t & 1) * 1024, sbn + (kb(t >> 1) << 11), AUX);
  #pragma unroll
  for (int k = 0; k < 4; ++k) wq[k] = *(LAS v4u*)(wl + k * 1024 + lane * 16);

  LAS char* rbw[4];
  #pragma unroll
  for (int k = 0; k < 4; ++k) rbw[k] = wl + 32768 + m * 512 + (((4 * k + g) ^ m) << 4);
  const int rr = 4 * wave + (lane >> 5);
  LAS char* rbr[2][2];
  #pragma unroll
  for (int hf = 0; hf < 2; ++hf) {
    rbr[hf][0] = (LAS char*)smem + 32768 + (rr + 2 * hf) * 512 + (((lane & 31) ^ (rr + 2 * hf)) << 4);
    rbr[hf][1] = rbr[hf][0] + 81920;
  }
  const unsigned ooff = (rr * NS + cg * 128 + (lane & 31) * 4) * 4;
  v4f acc[8];
#define PGAP 22
  auto step = [&](int n, bool prev, bool cur) __attribute__((always_inline)) {
    const unsigned sbc = sbn;
    unsigned fl = 0;
    v4f s0, s1, s2, s3;
    auto pw = [&](int c) { *(LAS v4f*)(rbw[c & 3] + (c >> 2) * 256) = acc[c]; };
    auto ldx = [&](int f, int half) {
      xb[2 * (f & 3) + half] = __builtin_amdgcn_raw_buffer_load_b128(rs, voff + half * 1024, (f < 8 ? sbc : sbn) + (kb(f & 7) << 11), AUX);
    };
    auto ldw = [&](int k, int t, int c) {
      wq[2 * k] = *(LAS v4u*)(wl + t * 4096 + c * 2048 + lane * 16);
      wq[2 * k + 1] = *(LAS v4u*)(wl + t * 4096 + c * 2048 + 1024 + lane * 16);
    };
    sfor<64>([&](auto G) __attribute__((always_inline)) {
      constexpr int gap = decltype(G)::value;
      constexpr bool pr = gap >= 48;
      constexpr int u = (gap - 48) >> 1;
      constexpr int t = pr ? 6 + (gap & 1) : gap >> 3, c = pr ? (u == 0 ? 0 : u < 4 ? u + 1 : u == 4 ? 1 : u) : gap & 7, k = pr ? gap & 1 : c & 1;
      if (prev && !cur && gap == 0) asm volatile("s_nop 7\n\ts_nop 7" ::: "memory");
      if (cur) {
        const v8i x = cat(xb[2 * (t & 3)], xb[2 * (t & 3) + 1]);
        if constexpr (c < 2) { const v8i w = cat(wq[2 * k], wq[2 * k + 1]); if (t == 0) mfma0v(acc[c], w, x); else mfmav(acc[c], w, x); }
        else if constexpr (c < 4) { if (t == 0) mfma0v(acc[c], Wv[t][c - 2], x); else mfmav(acc[c], Wv[t][c - 2], x); }
        else { if (t == 0) mfma0(acc[c], Wa[t][c - 4], x); else mfma(acc[c], Wa[t][c - 4], x); }
      }
      SB;
      if (cur) {
        if (gap < 40) { if (c == 2 || c == 3) ldw(c - 2, t + 1, c - 2); }
        else { if (gap == 42) ldw(0, 6, 0); if (gap == 43) ldw(1, 7, 0); if (gap == 50) ldw(0, 6, 1); if (gap == 52) ldw(1, 7, 1);
               if (gap == 58) ldw(0, 0, 0); if (gap == 60) ldw(1, 0, 1); }
        if (gap == PGAP) fl = __builtin_amdgcn_raw_buffer_load_b32(rf, 0, n + 1 < npw ? 4 : 0, AUX);
        if (gap == 38) {
          const unsigned target = __builtin_amdgcn_readlane(t_f, n + 1);
          sbn = __builtin_amdgcn_readlane(t_x, n + 1);
          if ((unsigned)__builtin_amdgcn_readfirstlane(fl) < target && alive) {
            unsigned f;
            if (!spin_ge(flagp + (n + 1 < npw ? 4 : 0), target, f, wsl, call)) { alive = false; scale = __builtin_nanf(""); }
            STAMP(8 + (n + 1 - nown) * 2 + hp + 1);
          }
        }
        if (gap < 48 && c == 7) ldx(t + 4, 0);
        if (gap >= 8 && gap <= 48 && (gap & 7) == 0) ldx((gap >> 3) + 3, 1);
        if (gap == 62) ldx(10, 0);
        if (prev) { if (gap == 0) ldx(2, 1); if (gap == 4) ldx(3, 0); if (gap == 5) ldx(3, 1); }
        if (gap == 51) pw(0); if (gap == 53) pw(2); if (gap == 55) pw(3); if (gap == 57) pw(4);
        if (gap == 59) pw(1); if (gap == 61) pw(5); if (gap == 63) pw(6);
      }
      if (prev) {
        if (gap == 1) pw(7);
        if (gap == 6) { asm volatile("s_waitcnt lgkmcnt(0)" ::: "memory"); __builtin_amdgcn_s_barrier(); asm volatile("" ::: "memory"); }
        #pragma unroll
        for (int hf = 0; hf < 2; ++hf) {
          if (gap == (hf ? 28 : 17)) { s0 = *(LAS v4f*)(rbr[hf][0]); s1 = *(LAS v4f*)(rbr[hf][0] + 40960); }
          if (gap == (hf ? 29 : 20)) { s2 = *(LAS v4f*)(rbr[hf][1]); s3 = *(LAS v4f*)(rbr[hf][1] + 40960); }
          if (gap == (hf ? 33 : 21)) { for (int e = 0; e < 4; ++e) asm("v_add_f32 %0, %1, %2" : "=v"(s0[e]) : "v"(s0[e]), "v"(s1[e])); }
          if (gap == (hf ? 34 : 24)) { for (int e = 0; e < 4; ++e) asm("v_add_f32 %0, %1, %2" : "=v"(s2[e]) : "v"(s2[e]), "v"(s3[e])); }
          if (gap == (hf ? 35 : 25)) { for (int e = 0; e < 4; ++e) asm("v_add_f32 %0, %1, %2" : "=v"(s0[e]) : "v"(s0[e]), "v"(s2[e])); }
          if (gap == (hf ? 36 : 26)) { for (int e = 0; e < 4; ++e) asm("v_mul_f32 %0, %1, %2" : "=v"(s0[e]) : "s"(scale), "v"(s0[e])); }
          if (gap == (hf ? 37 : 27)) __builtin_amdgcn_raw_buffer_store_b128(__builtin_bit_cast(v4u, s0), ro, ooff, __builtin_amdgcn_readlane(t_o, n - 1) + hf * 2 * NS * 4, 0);
        }
        if (gap == 46 && cur) { __builtin_amdgcn_s_barrier(); asm volatile("" ::: "memory"); }
      }
      SB;
    });
  };
  step(0, false, true);
  STAMP(4);
  for (int n = 1; n < np; ++n) {
    step(n, true, true);
    STAMP(43 + n);
  }
  step(np, true, false);
  STAMP(3);
  STAMP_OUT(0);
}

}

extern "C" void kernel_launch(int rank, int nranks, ncclComm_t comm, hipStream_t stream,
                              void* const* d_in, const int* in_sizes, int n_in,
                              void* d_out, int out_size,
                              const int* in_dtypes, int out_dtype,
                              void* const* d_out_all,
                              void* const* d_ws_all, size_t ws_bytes) {
  (void)comm; (void)in_dtypes; (void)out_dtype; (void)d_out_all;
  if (nranks != NR || n_in != 4 || (unsigned)rank >= (unsigned)NR || ws_bytes < WS_USED ||
      in_sizes[0] != MS * K || in_sizes[1] != K * NFULL || out_size != M * NS) {
    fprintf(stderr, "kernel_launch: unexpected problem shape\n");
    return;
  }
  HIP_CHECK(hipFuncSetAttribute((const void*)fused_ag_gemm, hipFuncAttributeMaxDynamicSharedMemorySize, LDS_BYTES));
  Params P;
  int order[NR][NRG];
  for (int r = 0; r < NR; ++r) {
    int n = 0;
    for (int same = 1; same >= 0; --same)
      for (int k = 1; k < NR; ++k) {
        const int p = (r + k) % NR;
        if (((((GROUP_MASK >> p) ^ (GROUP_MASK >> r)) & 1) == 0) == (same == 1)) order[r][n++] = p;
      }
  }
  P.peers = 0; P.back = 0;
  for (int i = 0; i < NRG; ++i) {
    const int p = order[rank][i];
    int bk = 0;
    for (int k = 0; k < NRG; ++k) if (order[p][k] == rank) bk = k;
    P.ws[i] = (char*)d_ws_all[p];
    P.peers |= (unsigned)p << (4 * i);
    P.back |= (unsigned)bk << (4 * i);
  }
  P.ws[NRG] = (char*)d_ws_all[rank];
  P.wsl = (char*)d_ws_all[rank];
  P.x = (const char*)d_in[0];
  P.w = (const char*)d_in[1];
  P.sx = (const float*)d_in[2];
  P.sw = (const float*)d_in[3];
  P.out = (float*)d_out;
  P.rank = rank;
  P.pad = 0;
  fused_ag_gemm<<<NWG, 256, LDS_BYTES, stream>>>(P);
  HIP_CHECK(hipGetLastError());
}
```
